```python
import numpy as np

kernel_name = "gpu_ag_gemm_m2048_k2048_n2048_f32_silu"


ABSMAX_RTOL = 0.001

COMPUTE_DTYPE = "float32"
EPILOGUE = "silu"
IN_QUANT = None
OUT_QUANT = None
IN_DTYPES = {"x": "float32", "w_mat": "float32"}
WORLD_SIZE = 8

def setup_inputs():

    x = np.random.randn(2048, 2048).astype(np.float32)
    w_mat = (np.random.randn(2048, 2048) / np.sqrt(2048)).astype(np.float32)
    return {"x": x, "w_mat": w_mat}

def _epilogue(y):
    return (y / (1.0 + np.exp(-np.clip(y, -60.0, 60.0)))).astype(np.float32)

def reference(x, w_mat):
    y = (x @ w_mat).astype(np.float32)
    return _epilogue(y)
```

```cpp
#include <hip/hip_runtime.h>
#include <rccl/rccl.h>
#include <cstdio>

#define HIP_CHECK(expr) do { hipError_t _e = (expr); if (_e != hipSuccess) { \
    fprintf(stderr, "HIP error %s at %s:%d\n", hipGetErrorString(_e), __FILE__, __LINE__); \
    return; } } while (0)

typedef float f4 __attribute__((ext_vector_type(4)));
typedef unsigned u4 __attribute__((ext_vector_type(4)));
typedef _Float16 h8 __attribute__((ext_vector_type(8)));
typedef __attribute__((address_space(1))) unsigned gu32;
typedef __attribute__((address_space(3))) unsigned lu32;

constexpr int NR = 8;
constexpr int KD = 2048;
constexpr int MS = 256;
constexpr int NL = 256;
constexpr int KSTEPS = KD / 32;
constexpr unsigned SLOT = MS * KD * 2;
constexpr unsigned XBUF = NR * SLOT;
constexpr unsigned CTRL_OFF = 2 * XBUF;
constexpr unsigned CTR_OFF = CTRL_OFF;
constexpr unsigned ZFLAG_OFF = CTRL_OFF + 256 * 64;
constexpr unsigned TMO_OFF = ZFLAG_OFF + 1024;
constexpr unsigned WS_NEED = TMO_OFF + 256;
constexpr int GRID = 256;
constexpr int NPROD = 64;
constexpr int NZW = GRID - NPROD;
constexpr int LDS_B = KD * 32 * 2;
constexpr int LDS_BYTES = LDS_B + 64;
constexpr unsigned long long DEADLINE = 50000000ull;
#define PUSH_WIN 7

struct Params {
    const float* x;
    const float* w;
    float* out;
    char* ws[NR];
    int rank;
};

__device__ __forceinline__ unsigned ld_agent(const char* p) {
    return __hip_atomic_load((gu32*)p, __ATOMIC_RELAXED, __HIP_MEMORY_SCOPE_AGENT);
}
__device__ __forceinline__ unsigned ld_sys(const char* p) {
    return __hip_atomic_load((gu32*)p, __ATOMIC_RELAXED, __HIP_MEMORY_SCOPE_SYSTEM);
}
__device__ __forceinline__ void st_agent(char* p, unsigned v) {
    __hip_atomic_store((gu32*)p, v, __ATOMIC_RELAXED, __HIP_MEMORY_SCOPE_AGENT);
}

__device__ __forceinline__ h8 cvt8(f4 a, f4 b) {
    h8 h;
    h[0] = (_Float16)a[0]; h[1] = (_Float16)a[1]; h[2] = (_Float16)a[2]; h[3] = (_Float16)a[3];
    h[4] = (_Float16)b[0]; h[5] = (_Float16)b[1]; h[6] = (_Float16)b[2]; h[7] = (_Float16)b[3];
    return h;
}

__device__ __forceinline__ float silu(float y) {
    float c = fminf(fmaxf(y, -60.f), 60.f);
    float e = __expf(-c);
    return y * __builtin_amdgcn_rcpf(1.f + e);
}

__global__ void __launch_bounds__(320, 2) ag_gemm_kernel(Params p) {
    extern __shared__ __attribute__((aligned(16))) char smem[];
    const int lane = threadIdx.x & 63;
    const int wave = __builtin_amdgcn_readfirstlane(threadIdx.x >> 6);
    const int bid = blockIdx.x;
    const int rank = p.rank;
    char* myws = p.ws[rank];
    lu32* lds_cnt = (lu32*)(smem + LDS_B);

    char* ctrp = myws + CTR_OFF + bid * 64 + wave * 4;
    const unsigned long long t_start = __builtin_amdgcn_s_memrealtime();

    if (wave == 4) {
        if (bid < NPROD) {
            const int rt = bid >> 2, ph = bid & 3;
            const float* src = p.x + (size_t)(rt * 16 + (lane & 15)) * KD + ph * 32 + (lane >> 4) * 8;
            f4 xa[16], xb[16];
            const unsigned callv = ld_agent(ctrp);
            #pragma unroll
            for (int i = 0; i < 16; ++i) {
                xa[i] = *(const f4*)(src + i * 128);
                xb[i] = *(const f4*)(src + i * 128 + 4);
            }
            __builtin_amdgcn_s_barrier();
            const unsigned call = __builtin_amdgcn_readfirstlane(callv) + 1u;
            if (lane == 0) st_agent(ctrp, call);
            const unsigned boff = (call & 1u) * XBUF + rank * SLOT;
            __amdgpu_buffer_rsrc_t rs[7];
            const int rot = bid % 7;
            #pragma unroll
            for (int d = 0; d < 7; ++d) {
                int j = d + rot; if (j >= 7) j -= 7;
                const int peer = (rank + 1 + j) & 7;
                rs[d] = __builtin_amdgcn_make_buffer_rsrc(p.ws[peer] + boff, 0, SLOT, 0x00020000);
            }
            const unsigned voff = (unsigned)((rt * KSTEPS + ph) * 1024 + lane * 16);
            unsigned zf0 = 0, zf1 = 0, zf2 = 0;
            bool ok = true;
            #pragma unroll
            for (int i = 0; i < 16; ++i) {
                u4 v = __builtin_bit_cast(u4, cvt8(xa[i], xb[i]));
                #pragma unroll
                for (int e = 0; e < 4; ++e) v[e] = v[e] == 0u ? 0x80000000u : v[e];
                if (i == 10) {
                    zf0 = ld_agent(myws + ZFLAG_OFF + lane * 4);
                    zf1 = ld_agent(myws + ZFLAG_OFF + 256 + lane * 4);
                    zf2 = ld_agent(myws + ZFLAG_OFF + 512 + lane * 4);
                }
                if (i == 15) {
                    while (!__all(zf0 == call && zf1 == call && zf2 == call)) {
                        if (__builtin_amdgcn_readfirstlane(ld_agent(myws + TMO_OFF)) == call ||
                            __builtin_amdgcn_s_memrealtime() - t_start > DEADLINE) {
                            if (lane == 0) st_agent(myws + TMO_OFF, call);
                            ok = false;
                            break;
                        }
                        __builtin_amdgcn_s_sleep(2);
                        zf0 = ld_agent(myws + ZFLAG_OFF + lane * 4);
                        zf1 = ld_agent(myws + ZFLAG_OFF + 256 + lane * 4);
                        zf2 = ld_agent(myws + ZFLAG_OFF + 512 + lane * 4);
                    }
                    if (!ok) break;
                }
                #pragma unroll
                for (int d = 0; d < 7; ++d)
                    __builtin_amdgcn_raw_buffer_store_b128(v, rs[d], voff + i * 4096, 0, 17);
                asm volatile("s_waitcnt vmcnt(%0)" :: "i"(PUSH_WIN) : "memory");
            }
        } else {
            const unsigned callv = ld_agent(ctrp);
            __builtin_amdgcn_s_barrier();
            const unsigned call = __builtin_amdgcn_readfirstlane(callv) + 1u;
            if (lane == 0) st_agent(ctrp, call);
            const int z = bid - NPROD;
            __amdgpu_buffer_rsrc_t rz = __builtin_amdgcn_make_buffer_rsrc(myws + ((call + 1u) & 1u) * XBUF, 0, XBUF, 0x00020000);
            const u4 zero = {0u, 0u, 0u, 0u};
            for (int u = z; u < 7 * 1024; u += NZW) {
                int s = u >> 10; s += (s >= rank);
                __builtin_amdgcn_raw_buffer_store_b128(zero, rz, (unsigned)(s * SLOT + (u & 1023) * 1024 + lane * 16), 0, 17);
            }
            asm volatile("s_waitcnt vmcnt(0)" ::: "memory");
            if (lane == 0) st_agent(myws + ZFLAG_OFF + z * 4, call);
        }
        return;
    }

    const int src = bid & 7;
    const int rb = (bid >> 3) & 3;
    const int cb = bid >> 5;
    const int n0 = cb * 32;
    const unsigned callv = ld_agent(ctrp);
    if (threadIdx.x == 0) __hip_atomic_store(lds_cnt, 0u, __ATOMIC_RELAXED, __HIP_MEMORY_SCOPE_WORKGROUP);
    asm volatile("s_waitcnt lgkmcnt(0)" ::: "memory");
    __builtin_amdgcn_s_barrier();

    {
        const int q = lane & 7, kgl = lane >> 3;
        const int t = q >> 2;
        #pragma unroll
        for (int it = 0; it < 2; ++it) {
            f4 r[4][8];
            #pragma unroll
            for (int b = 0; b < 4; ++b) {
                const int kg = ((it * 4 + b) * 4 + wave) * 8 + kgl;
                const float* wp = p.w + (size_t)(kg * 8) * NL + n0 + q * 4;
                #pragma unroll
                for (int j = 0; j < 8; ++j) r[b][j] = *(const f4*)(wp + j * NL);
            }
            #pragma unroll
            for (int b = 0; b < 4; ++b) {
                const int kg = ((it * 4 + b) * 4 + wave) * 8 + kgl;
                const int g = kg & 3, s = kg >> 2;
                #pragma unroll
                for (int i = 0; i < 4; ++i) {
                    h8 h;
                    #pragma unroll
                    for (int j = 0; j < 8; ++j) h[j] = (_Float16)r[b][j][i];
                    const int c = (4 * (q & 3) + i) ^ (q >> 1);
                    *(h8*)(smem + ((s * 2 + t) * 64 + g * 16 + c) * 16) = h;
                }
            }
        }
    }
    const unsigned call = __builtin_amdgcn_readfirstlane(callv) + 1u;
    if (lane == 0) st_agent(ctrp, call);
    asm volatile("s_waitcnt lgkmcnt(0)" ::: "memory");
    if (lane == 0) __hip_atomic_fetch_add(lds_cnt, 1u, __ATOMIC_RELAXED, __HIP_MEMORY_SCOPE_WORKGROUP);
    while (__hip_atomic_load(lds_cnt, __ATOMIC_RELAXED, __HIP_MEMORY_SCOPE_WORKGROUP) < 4u) __builtin_amdgcn_s_sleep(1);
    asm volatile("" ::: "memory");

    const int c = lane & 15, g = lane >> 4;
    const char* b0p = smem + (g * 16 + (c ^ (c >> 3))) * 16;
    const char* b1p = smem + 1024 + (g * 16 + (c ^ ((c >> 3) | 2))) * 16;
    f4 acc0 = {0.f, 0.f, 0.f, 0.f}, acc1 = {0.f, 0.f, 0.f, 0.f};
    const int rt = rb * 4 + wave;
    bool bad = false;

    if (src == rank) {
        const float* xp = p.x + (size_t)(rt * 16 + c) * KD + g * 8;
        #pragma unroll 1
        for (int grp = 0; grp < 8; ++grp) {
            f4 xa[8], xb[8];
            #pragma unroll
            for (int s = 0; s < 8; ++s) {
                xa[s] = *(const f4*)(xp + (grp * 8 + s) * 32);
                xb[s] = *(const f4*)(xp + (grp * 8 + s) * 32 + 4);
            }
            #pragma unroll
            for (int s = 0; s < 8; ++s) {
                const h8 a = cvt8(xa[s], xb[s]);
                const h8 w0 = *(const h8*)(b0p + (grp * 8 + s) * 2048);
                const h8 w1 = *(const h8*)(b1p + (grp * 8 + s) * 2048);
                acc0 = __builtin_amdgcn_mfma_f32_16x16x32_f16(w0, a, acc0, 0, 0, 0);
                acc1 = __builtin_amdgcn_mfma_f32_16x16x32_f16(w1, a, acc1, 0, 0, 0);
            }
        }
    } else {
        const char* xbase = myws + (call & 1u) * XBUF + src * SLOT + rt * (KSTEPS * 1024);
        __amdgpu_buffer_rsrc_t rx = __builtin_amdgcn_make_buffer_rsrc((void*)xbase, 0, KSTEPS * 1024, 0x00020000);
        bool optimistic = true;
        #pragma unroll 1
        for (int grp = 0; grp < 8; ++grp) {
            u4 a[8];
            auto load_group = [&]() -> bool {
                #pragma unroll
                for (int s = 0; s < 8; ++s)
                    a[s] = __builtin_amdgcn_raw_buffer_load_b128(rx, (unsigned)(lane * 16), (grp * 8 + s) * 1024, 17);
                unsigned m = 0xffffffffu;
                #pragma unroll
                for (int s = 0; s < 8; ++s) m = min(m, min(min(a[s][0], a[s][1]), min(a[s][2], a[s][3])));
                return __all(m != 0u);
            };
            bool have = false;
            if (optimistic) have = load_group();
            if (!have) {
                const bool probed_first = !optimistic;
                const char* pp = xbase + (grp * 8 + 4 + (lane & 3)) * 1024 + 1020;
                int tries = 0;
                do {
                    while (!__all(ld_sys(pp) != 0u)) {
                        ++tries;
                        if ((tries & 15) == 0) {
                            if (__builtin_amdgcn_readfirstlane(ld_agent(myws + TMO_OFF)) == call ||
                                __builtin_amdgcn_s_memrealtime() - t_start > DEADLINE) {
                                if (lane == 0) st_agent(myws + TMO_OFF, call);
                                bad = true;
                                break;
                            }
                        }
                        __builtin_amdgcn_s_sleep(1);
                    }
                    if (bad) break;
                    have = load_group();
                    if (!have) {
                        tries += 16 - (tries & 15) - 1;
                        if (__builtin_amdgcn_s_memrealtime() - t_start > DEADLINE) { if (lane == 0) st_agent(myws + TMO_OFF, call); bad = true; break; }
                    }
                } while (!have);
                optimistic = probed_first && tries == 0;
            }
            if (bad) break;
            #pragma unroll
            for (int s = 0; s < 8; ++s) {
                const h8 av = __builtin_bit_cast(h8, a[s]);
                const h8 w0 = *(const h8*)(b0p + (grp * 8 + s) * 2048);
                const h8 w1 = *(const h8*)(b1p + (grp * 8 + s) * 2048);
                acc0 = __builtin_amdgcn_mfma_f32_16x16x32_f16(w0, av, acc0, 0, 0, 0);
                acc1 = __builtin_amdgcn_mfma_f32_16x16x32_f16(w1, av, acc1, 0, 0, 0);
            }
        }
    }
    f4 o0, o1;
    #pragma unroll
    for (int r = 0; r < 4; ++r) { o0[r] = silu(acc0[r]); o1[r] = silu(acc1[r]); }
    if (bad) {
        const float qn = __builtin_nanf("");
        o0 = f4{qn, qn, qn, qn}; o1 = o0;
    }
    float* op = p.out + (size_t)(src * MS + rt * 16 + c) * NL + n0 + g * 4;
    *(f4*)op = o0;
    *(f4*)(op + 16) = o1;
}

extern "C" void kernel_launch(int rank, int nranks, ncclComm_t comm,
                              hipStream_t stream,
                              void* const* d_in, const int* in_sizes, int n_in,
                              void* d_out, int out_size,
                              const int* in_dtypes, int out_dtype,
                              void* const* d_out_all,
                              void* const* d_ws_all, size_t ws_bytes) {
    if (nranks != NR || ws_bytes < WS_NEED || n_in < 2) { fprintf(stderr, "unexpected configuration\n"); return; }
    static bool init = false;
    if (!init) {
        HIP_CHECK(hipFuncSetAttribute((const void*)ag_gemm_kernel, hipFuncAttributeMaxDynamicSharedMemorySize, LDS_BYTES));
        init = true;
    }
    Params p;
    p.x = (const float*)d_in[0];
    p.w = (const float*)d_in[1];
    p.out = (float*)d_out;
    for (int i = 0; i < NR; ++i) p.ws[i] = (char*)d_ws_all[i];
    p.rank = rank;
    hipLaunchKernelGGL(ag_gemm_kernel, dim3(GRID), dim3(320), LDS_BYTES, stream, p);
    HIP_CHECK(hipGetLastError());
}
```

```cpp
#include <hip/hip_runtime.h>
#include <rccl/rccl.h>
#include <cstdio>
#include <type_traits>

#define HIP_CHECK(expr) do { hipError_t _e = (expr); if (_e != hipSuccess) { \
    fprintf(stderr, "HIP error %s at %s:%d\n", hipGetErrorString(_e), __FILE__, __LINE__); \
    return; } } while (0)

typedef float f4 __attribute__((ext_vector_type(4)));
typedef unsigned u4 __attribute__((ext_vector_type(4)));
typedef _Float16 h8 __attribute__((ext_vector_type(8)));
typedef __attribute__((address_space(1))) unsigned gu32;
typedef __attribute__((address_space(3))) unsigned lu32;

#define N1 12
constexpr int N2 = 16 - N1;
constexpr int NR = 8;
constexpr int KD = 2048;
constexpr int MS = 256;
constexpr int NL = 256;
constexpr unsigned TILE_B = 16 * KD * 2;
constexpr unsigned X_BYTES = 64 * TILE_B;
constexpr unsigned IMG_B = KD * 32 * 2;
constexpr unsigned W_OFF = X_BYTES;
constexpr unsigned PAR = X_BYTES + 8 * IMG_B;
constexpr unsigned CTRL_OFF = 2 * PAR;
constexpr unsigned CTR_OFF = CTRL_OFF;
constexpr unsigned ZFLAG_OFF = CTRL_OFF + 256 * 64;
constexpr unsigned PFLAG_OFF = ZFLAG_OFF + 256;
constexpr unsigned TMO_OFF = PFLAG_OFF + 256;
constexpr unsigned WS_NEED = TMO_OFF + 256;
constexpr unsigned OUT_BYTES = 2048u * NL * 4;
constexpr unsigned SENT = 0xFFFFFFFFu;
constexpr unsigned QNAN = 0x7FC00000u;
constexpr int GRID = 256;
constexpr int BLOCK = 640;
constexpr unsigned PART_B = 8 * 16384;
constexpr int LDS_BYTES = PART_B + 128;
constexpr unsigned DEADLINE = 50000000u;
#define IWIN 4
#define XWIN 4
#define ZW 2
#define ZWIN 2
#define PSLEEP 2
#define WSLEEP 4

#define STAMP(i) do {} while (0)
#define PDUMP() do {} while (0)

struct Params {
    const float* x;
    const float* w;
    float* out;
    char* myws;
    char* mates[6];
    char* opps[8];
    float* oout[4];
    int rank;
};

__device__ __forceinline__ unsigned uni(unsigned v) { return __builtin_amdgcn_readfirstlane(v); }
__device__ __forceinline__ unsigned now32() { return (unsigned)__builtin_amdgcn_s_memrealtime(); }
__device__ __forceinline__ unsigned ld_agent(const char* p) {
    return __hip_atomic_load((gu32*)p, __ATOMIC_RELAXED, __HIP_MEMORY_SCOPE_AGENT);
}
__device__ __forceinline__ unsigned ld_sys(const char* p) {
    return __hip_atomic_load((gu32*)p, __ATOMIC_RELAXED, __HIP_MEMORY_SCOPE_SYSTEM);
}
__device__ __forceinline__ void st_agent(char* p, unsigned v) {
    __hip_atomic_store((gu32*)p, v, __ATOMIC_RELAXED, __HIP_MEMORY_SCOPE_AGENT);
}
__device__ __forceinline__ unsigned lds_ld(lu32* a) { return __hip_atomic_load(a, __ATOMIC_RELAXED, __HIP_MEMORY_SCOPE_WORKGROUP); }
__device__ __forceinline__ void lds_st(lu32* a, unsigned v) { __hip_atomic_store(a, v, __ATOMIC_RELAXED, __HIP_MEMORY_SCOPE_WORKGROUP); }
__device__ __forceinline__ __amdgpu_buffer_rsrc_t mkrs(const void* p, unsigned bytes) {
    const unsigned long long a = (unsigned long long)p;
    const unsigned lo = uni((unsigned)a), hi = uni((unsigned)(a >> 32));
    return __builtin_amdgcn_make_buffer_rsrc((void*)(((unsigned long long)hi << 32) | lo), 0, bytes, 0x00020000);
}
__device__ __forceinline__ f4 bld(__amdgpu_buffer_rsrc_t r, unsigned vo, unsigned so) {
    return __builtin_bit_cast(f4, __builtin_amdgcn_raw_buffer_load_b128(r, vo, so, 0));
}
__device__ __forceinline__ u4 cld(__amdgpu_buffer_rsrc_t r, unsigned vo, unsigned so) {
    return __builtin_amdgcn_raw_buffer_load_b128(r, vo, so, 17);
}
__device__ __forceinline__ void cst(u4 v, __amdgpu_buffer_rsrc_t r, unsigned vo, unsigned so) {
    __builtin_amdgcn_raw_buffer_store_b128(v, r, vo, so, 17);
}
__device__ __forceinline__ u4 nz(u4 v) {
    #pragma unroll
    for (int e = 0; e < 4; ++e) v[e] = v[e] == 0u ? 0x80000000u : v[e];
    return v;
}
__device__ __forceinline__ u4 cvt8(f4 a, f4 b) {
    h8 h;
    h[0] = (_Float16)a[0]; h[1] = (_Float16)a[1]; h[2] = (_Float16)a[2]; h[3] = (_Float16)a[3];
    h[4] = (_Float16)b[0]; h[5] = (_Float16)b[1]; h[6] = (_Float16)b[2]; h[7] = (_Float16)b[3];
    return nz(__builtin_bit_cast(u4, h));
}
__device__ __forceinline__ unsigned min4(u4 v) { return min(min(v[0], v[1]), min(v[2], v[3])); }
__device__ __forceinline__ unsigned max4(u4 v) { return max(max(v[0], v[1]), max(v[2], v[3])); }
__device__ __forceinline__ bool all_set(bool b) { return __ballot(b) == ~0ull; }

__device__ __forceinline__ float silu(float y) {
    float c = fminf(fmaxf(y, -60.f), 60.f);
    float e = __expf(-c);
    return y * __builtin_amdgcn_rcpf(1.f + e);
}

__device__ __forceinline__ bool expired(char* myws, unsigned call, unsigned t0, unsigned& tries) {
    tries = uni(tries + 1u);
    if ((tries & 31u) != 0u) return false;
    if (uni(ld_agent(myws + TMO_OFF)) == call || now32() - t0 > DEADLINE) {
        st_agent(myws + TMO_OFF, call);
        return true;
    }
    return false;
}
__device__ __forceinline__ bool wait_flags(char* myws, unsigned off, unsigned v, unsigned call, unsigned t0, int lane) {
    unsigned tries = 0;
    while (!all_set(v == call)) {
        if (expired(myws, call, t0, tries)) return false;
        __builtin_amdgcn_s_sleep(2);
        v = ld_agent(myws + off + lane * 4);
    }
    return true;
}

__global__ void __launch_bounds__(BLOCK, 3) ag_gemm_kernel(Params p) {
    extern __shared__ __attribute__((aligned(16))) char smem[];
    const int lane = threadIdx.x & 63;
    const int wave = uni(threadIdx.x >> 6);
    const int bid = blockIdx.x;
    const int rank = p.rank;
    const int gr = rank >> 2, li = rank & 3;
    char* myws = p.myws;

    char* ctrp = myws + CTR_OFF + bid * 64 + wave * 4;
    const unsigned t0 = now32();
    STAMP(0);

    if (bid < 32) {
        if (wave != 0) return;
        const int j = bid;
        __amdgpu_buffer_rsrc_t rxin = mkrs(p.x, MS * KD * 4);
        const unsigned xv = (unsigned)(((lane & 15) * KD + (lane >> 4) * 8 + j * 32) * 4);
        constexpr int NU = 2 * N1, PB = 12;
        f4 xa[PB], xb[PB];
        const unsigned callv = ld_agent(ctrp);
        #pragma unroll
        for (int u = 0; u < PB; ++u) {
            xa[u] = bld(rxin, xv, (u >> 1) * 16 * KD * 4 + (u & 1) * 4096); xb[u] = bld(rxin, xv + 16, (u >> 1) * 16 * KD * 4 + (u & 1) * 4096);
        }
        __amdgpu_buffer_rsrc_t rs[4];
        const int rot = j % 3;
        #pragma unroll
        for (int d = 0; d < 3; ++d) rs[d] = mkrs(p.mates[rot + d], 2 * PAR);
        rs[3] = mkrs(myws, 2 * PAR);
        __builtin_amdgcn_sched_barrier(0);
        const unsigned call = uni(callv) + 1u;
        const unsigned boff = (call & 1u) * PAR + li * 16 * TILE_B + j * 1024;
        const unsigned voff = lane * 16;
        unsigned zf = 0;
        #pragma unroll
        for (int u = 0; u < NU; ++u) {
            const u4 v = cvt8(xa[u % PB], xb[u % PB]);
            if (u == NU - 6) zf = ld_agent(myws + ZFLAG_OFF + lane * 4);
            if (u == NU - 1) { if (!wait_flags(myws, ZFLAG_OFF, zf, call, t0, lane)) break; }
            #pragma unroll
            for (int d = 0; d < 4; ++d) cst(v, rs[d], voff, boff + (u >> 1) * TILE_B + (u & 1) * 32768);
            if (u + PB < NU) {
                const unsigned so = ((u + PB) >> 1) * 16 * KD * 4 + ((u + PB) & 1) * 4096;
                xa[u % PB] = bld(rxin, xv, so); xb[u % PB] = bld(rxin, xv + 16, so);
            }
            if (u >= 1) asm volatile("s_waitcnt vmcnt(%0)" :: "i"(IWIN + (u + PB - 1 < NU ? 2 : 0) + (u + PB < NU ? 2 : 0)) : "memory");
            __builtin_amdgcn_sched_barrier(0);
        }
        st_agent(ctrp, call);
        return;
    }
    if (bid < 64) {
        if (wave != 0) return;
        const int jj = bid - 32;
        const int dq = jj & 3, cb2 = (jj >> 2) & 1, kq = jj >> 3;
        const int q = lane & 7, kgl = lane >> 3;
        __amdgpu_buffer_rsrc_t rwin = mkrs(p.w, KD * NL * 4);
        const unsigned wv = (unsigned)(((kq * 64 + kgl * 8) * NL + dq * 64 + cb2 * 32 + q * 4) * 4);
        const unsigned callv = ld_agent(ctrp);
        constexpr int WB = 3;
        f4 r[WB][8];
        #pragma unroll
        for (int m = 0; m < WB; ++m)
            #pragma unroll
            for (int e = 0; e < 8; ++e) r[m][e] = bld(rwin, wv + (e & 3) * 1024, m * 262144 + (e >> 2) * 4096);
        __amdgpu_buffer_rsrc_t rs[4];
        #pragma unroll
        for (int d = 0; d < 4; ++d) rs[d] = mkrs(p.opps[dq + d], 2 * PAR);
        __builtin_amdgcn_sched_barrier(0);
        const unsigned call = uni(callv) + 1u;
        const unsigned par = (call & 1u) * PAR;
        const unsigned woff = par + W_OFF + (li * 2 + cb2) * IMG_B + kq * 4096;
        const unsigned voff = lane * 16;
        char* lw = smem + (((kgl >> 2) * 2 + (q >> 2)) * 64 + (kgl & 3) * 16 + 4 * (q & 3)) * 16;
        const int sw = q >> 1;
        const int lr0 = ((lane >> 4) * 16 + ((lane & 15) ^ ((lane >> 3) & 1))) * 16;
        const int lr1 = ((lane >> 4) * 16 + ((lane & 15) ^ (((lane >> 3) & 1) | 2))) * 16;
        __amdgpu_buffer_rsrc_t rxin = mkrs(p.x, MS * KD * 4);
        const unsigned xv = (unsigned)(((N1 * 16 + (lane & 15)) * KD + (lane >> 4) * 8 + jj * 32) * 4);
        f4 xa[2 * N2], xb[2 * N2];
        unsigned pf = 0, zf = 0;
        bool ok = true;
        auto nld = [](int m) { return m + WB < 8 ? 8 : (((m + WB - 8) * 4 + 4 <= 2 * N2) ? 8 : (((m + WB - 8) * 4 < 2 * N2) ? 2 * (2 * N2 - (m + WB - 8) * 4) : 0)); };
        #pragma unroll
        for (int m = 0; m < 8; ++m) {
            #pragma unroll
            for (int i = 0; i < 4; ++i) {
                h8 h;
                #pragma unroll
                for (int e = 0; e < 8; ++e) h[e] = (_Float16)r[m % WB][e][i];
                *(h8*)(lw + (m & 1) * 4096 + ((i ^ sw) * 16)) = h;
            }
            u4 v[4];
            #pragma unroll
            for (int u = 0; u < 4; ++u) v[u] = nz(*(const u4*)(smem + (m & 1) * 4096 + u * 1024 + ((u & 1) ? lr1 : lr0)));
            if (m == 7) { ok = wait_flags(myws, PFLAG_OFF, pf, call, t0, lane); if (!ok) break; }
            if (m >= 2) asm volatile("s_waitcnt vmcnt(%0)" :: "i"(XWIN + nld(m - 2) + nld(m - 1) + (m == 5 || m == 6 ? 1 : 0)) : "memory");
            #pragma unroll
            for (int u = 0; u < 4; ++u) cst(v[u], rs[0], voff, woff + m * 16384 + u * 1024);
            if (m + WB < 8) {
                #pragma unroll
                for (int e = 0; e < 8; ++e) r[m % WB][e] = bld(rwin, wv + (e & 3) * 1024, (m + WB) * 262144 + (e >> 2) * 4096);
            } else {
                #pragma unroll
                for (int e = 0; e < 4; ++e) {
                    const int u = (m + WB - 8) * 4 + e;
                    if (u < 2 * N2) {
                        xa[u] = bld(rxin, xv, (u >> 1) * 16 * KD * 4 + (u & 1) * 4096); xb[u] = bld(rxin, xv + 16, (u >> 1) * 16 * KD * 4 + (u & 1) * 4096);
                    }
                }
            }
            if (m == 4) pf = ld_agent(myws + PFLAG_OFF + lane * 4);
            __builtin_amdgcn_sched_barrier(0);
        }
        STAMP(2);
        if (ok) {
            const unsigned boff = par + (li * 16 + N1) * TILE_B + jj * 1024;
            #pragma unroll
            for (int u = 0; u < 2 * N2; ++u) {
                const u4 v = cvt8(xa[u], xb[u]);
                if (u == 2 * N2 - 4) zf = ld_agent(myws + ZFLAG_OFF + lane * 4);
                if (u == 2 * N2 - 1) { if (!wait_flags(myws, ZFLAG_OFF, zf, call, t0, lane)) break; }
                #pragma unroll
                for (int d = 0; d < 4; ++d) cst(v, rs[d], voff, boff + (u >> 1) * TILE_B + (u & 1) * 32768);
                asm volatile("s_waitcnt vmcnt(%0)" :: "i"(XWIN) : "memory");
                __builtin_amdgcn_sched_barrier(0);
            }
        }
        st_agent(ctrp, call);
        return;
    }
    if (bid < 128) {
        if (wave >= 8) return;
        const unsigned callv = ld_agent(ctrp);
        const int z = bid - 64;
        {
            const int pc = z * 8 + wave;
            const int l = pc >> 7, i = (pc >> 3) & 15, cb = pc & 7;
            const int srcr = ((i < N1) ? (gr ^ 1) : gr) * 4 + l;
            __amdgpu_buffer_rsrc_t ro = mkrs(p.out, OUT_BYTES);
            const unsigned o = (unsigned)((lane >> 3) * NL * 4 + (lane & 7) * 16);
            const unsigned so = (unsigned)(((srcr * MS + i * 16) * NL + cb * 32) * 4);
            const u4 sv = {SENT, SENT, SENT, SENT};
            cst(sv, ro, o, so);
            cst(sv, ro, o, so + 8 * NL * 4);
        }
        const unsigned call = uni(callv) + 1u;
        asm volatile("s_waitcnt vmcnt(0)" ::: "memory");
        __builtin_amdgcn_s_barrier();
        if (wave == 0) st_agent(myws + PFLAG_OFF + z * 4, call);
        st_agent(ctrp, call);
        STAMP(1);
        if (wave >= ZW) return;
        __amdgpu_buffer_rsrc_t rz = mkrs(myws + ((call + 1u) & 1u) * PAR + z * (PAR / 64), PAR / 64);
        const u4 zero = {0u, 0u, 0u, 0u};
        #pragma unroll 4
        for (int u = 0; u < 80 / ZW; ++u) {
            cst(zero, rz, (unsigned)(lane * 16), (unsigned)((u * ZW + wave) * 1024));
            asm volatile("s_waitcnt vmcnt(%0)" :: "i"(ZWIN) : "memory");
        }
        asm volatile("s_waitcnt vmcnt(0)" ::: "memory");
        __builtin_amdgcn_s_barrier();
        if (wave == 0) st_agent(myws + ZFLAG_OFF + z * 4, call);
        STAMP(2);
        PDUMP();
        return;
    }

    const int xg = bid & 7;
    const int cidx = (bid - 128) >> 3;
    const bool foreign = cidx < 8;
    const unsigned callv = ld_agent(ctrp);
    const int c = lane & 15, g = lane >> 4;
    lu32* cnt = (lu32*)(smem + PART_B);
    lu32* rdy = cnt + 8;
    lu32* abortp = cnt + 16;
    const int l = xg >> 1, tp = xg & 1;
    const int ncol = foreign ? li * 64 + (cidx & 1) * 32 : (cidx - 8) * 32;
    constexpr int M2a = (N1 + 1) >> 1, M2b = N1 >> 1;
    const int m2 = tp ? M2b : M2a;
    auto late = [&](unsigned& tries) -> bool {
        tries = uni(tries + 1u);
        if ((tries & 127u) != 0u) return false;
        return uni(lds_ld(abortp)) != 0u || now32() - t0 > DEADLINE + 1000000u;
    };

    if (wave >= 8) {
        if (wave == 8) lds_st(cnt + (lane & 31), 0u);
        asm volatile("s_waitcnt lgkmcnt(0)" ::: "memory");
        __builtin_amdgcn_s_barrier();
        const unsigned call = uni(callv) + 1u;
        st_agent(ctrp, call);
        if (wave == 8) {
            const char* xb = myws + (call & 1u) * PAR + (l * 16 + tp) * TILE_B + 1020;
            const bool hi = lane >= 32;
            const unsigned lo = (lane & 31) * 1024 + 32768;
            int ma = 0, mb = m2;
            bool sa = false, sb = false;
            unsigned ka = 0, kb = 0, tries = 0;
            while (ma < m2 || mb < 8) {
                const unsigned oa = (unsigned)min(ma, 7) * (2 * TILE_B), ob = (unsigned)min(mb, 7) * (2 * TILE_B);
                const unsigned off = (hi ? ob : oa) + ((hi ? sb : sa) ? lo : 0u);
                const unsigned long long bal = __ballot(ld_sys(xb + off) != 0u);
                bool idle = true;
                #pragma unroll
                for (int h = 0; h < 2; ++h) {
                    unsigned b = (unsigned)(bal >> (32 * h));
                    int& mm = h ? mb : ma; bool& ss = h ? sb : sa; unsigned& kk = h ? kb : ka;
                    if (mm < (h ? 8 : m2)) {
                        if (ss) {
                            idle = false;
                            b &= b >> 1; b &= b >> 2; b &= 0x11111111u;
                            b = (b | (b >> 3)) & 0x03030303u; b = (b | (b >> 6)) & 0x000F000Fu; b = (b | (b >> 12)) & 0xFFu;
                            if (b != kk) { kk = b; lds_st(rdy + mm, b); }
                            if (b == 0xFFu) { ++mm; ss = false; kk = 0; }
                        } else if (b & 1u) { ss = true; idle = false; }
                    }
                }
                if (expired(myws, call, t0, tries)) { lds_st(abortp, 1u); break; }
                if (idle) __builtin_amdgcn_s_sleep(PSLEEP);
            }
        } else {
            __amdgpu_buffer_rsrc_t ro = mkrs(foreign ? (const void*)p.oout[cidx >> 1] : (const void*)p.out, OUT_BYTES);
            unsigned pend = 0xFFu, tries = 0;
            unsigned bad = 0;
            while (pend) {
                unsigned mk = (unsigned)__ballot(lds_ld(cnt + (lane & 7)) == 8u) & pend;
                if (!mk) {
                    if (late(tries)) { bad = 1; mk = pend; }
                    else { __builtin_amdgcn_s_sleep(1); continue; }
                }
                asm volatile("" ::: "memory");
                const int m = __builtin_ctz(mk);
                pend &= ~(1u << m);
                f4 a0 = {0.f, 0.f, 0.f, 0.f}, a1 = {0.f, 0.f, 0.f, 0.f};
                #pragma unroll
                for (int v = 0; v < 8; ++v) {
                    a0 += *(const f4*)(smem + v * 16384 + m * 2048 + lane * 16);
                    a1 += *(const f4*)(smem + v * 16384 + m * 2048 + 1024 + lane * 16);
                }
                u4 o0, o1;
                #pragma unroll
                for (int r = 0; r < 4; ++r) {
                    o0[r] = __builtin_bit_cast(unsigned, silu(a0[r])); o1[r] = __builtin_bit_cast(unsigned, silu(a1[r]));
                    o0[r] = (bad || o0[r] == SENT) ? QNAN : o0[r];
                    o1[r] = (bad || o1[r] == SENT) ? QNAN : o1[r];
                }
                const int ti = 2 * m + tp;
                const int srcr = ((ti < N1) ? gr : (gr ^ 1)) * 4 + l;
                const unsigned o = (unsigned)((c * NL + g * 4) * 4);
                const unsigned so = (unsigned)(((srcr * MS + ti * 16) * NL + ncol) * 4);
                cst(o0, ro, o, so);
                cst(o1, ro, o + 64, so);
                if (bad) tries = 127u;
            }
        }
        return;
    }

    char* myreg = smem + wave * 16384;
    h8 w0[8], w1[8];
    unsigned call;
    bool bad = false;
    if (!foreign) {
        const int q = lane & 7, kgl = lane >> 3;
        __amdgpu_buffer_rsrc_t rwin = mkrs(p.w, KD * NL * 4);
        const unsigned wv = (unsigned)(((wave * 128 + kgl * 8) * NL + ncol + q * 4) * 4);
        f4 r[4][8];
        #pragma unroll
        for (int b = 0; b < 4; ++b) {
            #pragma unroll
            for (int j = 0; j < 8; ++j) r[b][j] = bld(rwin, wv + (j & 3) * 1024, ((b >> 1) * 1024 + (b & 1) * 64) * NL * 4 + (j >> 2) * 4096);
        }
        __builtin_amdgcn_s_barrier();
        char* lw = myreg + (((kgl >> 2) * 2 + (q >> 2)) * 64 + (kgl & 3) * 16 + 4 * (q & 3)) * 16;
        #pragma unroll
        for (int b = 0; b < 4; ++b) {
            #pragma unroll
            for (int i = 0; i < 4; ++i) {
                h8 h;
                #pragma unroll
                for (int j = 0; j < 8; ++j) h[j] = (_Float16)r[b][j][i];
                *(h8*)(lw + b * 4096 + ((i ^ (q >> 1)) * 16)) = h;
            }
        }
        const char* b0p = myreg + (g * 16 + (c ^ (c >> 3))) * 16;
        const char* b1p = myreg + 1024 + (g * 16 + (c ^ ((c >> 3) | 2))) * 16;
        #pragma unroll
        for (int s = 0; s < 8; ++s) { w0[s] = *(const h8*)(b0p + s * 2048); w1[s] = *(const h8*)(b1p + s * 2048); }
        call = uni(callv) + 1u;
    } else {
        __builtin_amdgcn_s_barrier();
        call = uni(callv) + 1u;
        const char* ib = myws + (call & 1u) * PAR + W_OFF + cidx * IMG_B + wave * 8192;
        __amdgpu_buffer_rsrc_t ri = mkrs(ib, 65536 + 8192);
        const char* pp = ib + (lane & 2) * 32768 + (lane & 1) * 4096 + 4092;
        unsigned tries = 0;
        STAMP(7);
        bool have;
        do {
            while (!all_set(ld_sys(pp) != 0u)) {
                if (expired(myws, call, t0, tries)) { bad = true; break; }
                __builtin_amdgcn_s_sleep(WSLEEP);
            }
            STAMP(5);
            unsigned m = 0xffffffffu;
            #pragma unroll
            for (int s = 0; s < 8; ++s) {
                const u4 a = cld(ri, (unsigned)(lane * 16 + (s & 1) * 2048), (s >> 2) * 65536 + ((s >> 1) & 1) * 4096);
                const u4 b = cld(ri, (unsigned)(lane * 16 + (s & 1) * 2048 + 1024), (s >> 2) * 65536 + ((s >> 1) & 1) * 4096);
                w0[s] = __builtin_bit_cast(h8, a); w1[s] = __builtin_bit_cast(h8, b);
                m = min(m, min(min4(a), min4(b)));
            }
            have = all_set(m != 0u);
            if (!have && expired(myws, call, t0, tries)) bad = true;
        } while (!have && !bad);
        if (bad) lds_st(abortp, 1u);
    }
    st_agent(ctrp, call);
    STAMP(1);

    const char* xbase = myws + (call & 1u) * PAR + (l * 16 + tp) * TILE_B + wave * 4096;
    __amdgpu_buffer_rsrc_t rx = mkrs(xbase, 15 * TILE_B);

    auto process = [&](auto nt_tag, int ma, int mb) -> unsigned {
        constexpr int NT = decltype(nt_tag)::value;
        u4 a[NT * 8];
        int so[2] = {(int)uni(ma * (2 * TILE_B)), (int)uni(mb * (2 * TILE_B))};
        asm volatile("" : "+s"(so[0]), "+s"(so[1]));
        #pragma unroll
        for (int t = 0; t < NT; ++t) {
            #pragma unroll
            for (int s = 0; s < 8; ++s) a[t * 8 + s] = cld(rx, (unsigned)(lane * 16 + (s & 3) * 1024), so[t] + (s >> 2) * 32768);
            __builtin_amdgcn_sched_barrier(0);
        }
        unsigned okm = 0;
        #pragma unroll
        for (int t = 0; t < NT; ++t) {
            f4 n0 = {0.f, 0.f, 0.f, 0.f}, n1 = {0.f, 0.f, 0.f, 0.f};
            unsigned mn = 0xffffffffu;
            #pragma unroll
            for (int s = 0; s < 8; ++s) {
                const h8 av = __builtin_bit_cast(h8, a[t * 8 + s]);
                n0 = __builtin_amdgcn_mfma_f32_16x16x32_f16(w0[s], av, n0, 0, 0, 0);
                n1 = __builtin_amdgcn_mfma_f32_16x16x32_f16(w1[s], av, n1, 0, 0, 0);
                mn = min(mn, min4(a[t * 8 + s]));
            }
            const int m = t ? mb : ma;
            *(f4*)(myreg + m * 2048 + lane * 16) = n0;
            *(f4*)(myreg + m * 2048 + 1024 + lane * 16) = n1;
            if (all_set(mn != 0u)) okm |= 1u << m;
            __builtin_amdgcn_sched_barrier(0);
        }
        return okm;
    };

    {
        unsigned pend = bad ? 0u : 0xFFu, tries = 0;
        while (pend) {
            unsigned mk = (unsigned)__ballot((lds_ld(rdy + (lane & 7)) >> wave) & 1u) & pend;
            if (late(tries)) { bad = true; break; }
            if (mk == 0u) { __builtin_amdgcn_s_sleep(1); continue; }
            const int ma = __builtin_ctz(mk); mk &= mk - 1;
            unsigned okm;
            if (mk) okm = process(std::integral_constant<int, 2>{}, ma, __builtin_ctz(mk));
            else okm = process(std::integral_constant<int, 1>{}, ma, ma);
            pend &= ~okm;
            asm volatile("s_waitcnt lgkmcnt(0)" ::: "memory");
            if (lane < 8 && ((okm >> lane) & 1u)) __hip_atomic_fetch_add(cnt + lane, 1u, __ATOMIC_RELAXED, __HIP_MEMORY_SCOPE_WORKGROUP);
        }
    }
    STAMP(2);
    if (!bad && !foreign) {
        const int ti = 2 * wave + tp;
        const int srcr = ((ti < N1) ? gr : (gr ^ 1)) * 4 + l;
        __amdgpu_buffer_rsrc_t ro = mkrs(p.out, OUT_BYTES);
        const unsigned om = (unsigned)(((lane >> 3) * NL) * 4 + (lane & 7) * 16);
        const unsigned so = (unsigned)((((srcr ^ 4) * MS + ti * 16) * NL + ncol) * 4);
        unsigned tries = 0;
        for (;;) {
            const u4 a = cld(ro, om, so);
            const u4 b = cld(ro, om, so + 8 * NL * 4);
            if (all_set(max(max4(a), max4(b)) != SENT)) break;
            if (expired(myws, call, t0, tries)) break;
            __builtin_amdgcn_s_sleep(2);
        }
    }
}

extern "C" void kernel_launch(int rank, int nranks, ncclComm_t comm,
                              hipStream_t stream,
                              void* const* d_in, const int* in_sizes, int n_in,
                              void* d_out, int out_size,
                              const int* in_dtypes, int out_dtype,
                              void* const* d_out_all,
                              void* const* d_ws_all, size_t ws_bytes) {
    if (nranks != NR || ws_bytes < WS_NEED || n_in < 2 || rank < 0 || rank >= NR) { fprintf(stderr, "unexpected configuration\n"); return; }
    static bool init = false;
    if (!init) {
        HIP_CHECK(hipFuncSetAttribute((const void*)ag_gemm_kernel, hipFuncAttributeMaxDynamicSharedMemorySize, LDS_BYTES));
        init = true;
    }
    Params p;
    const int gr = rank >> 2, li = rank & 3;
    p.x = (const float*)d_in[0];
    p.w = (const float*)d_in[1];
    p.out = (float*)d_out;
    p.myws = (char*)d_ws_all[rank];
    for (int j = 0; j < 6; ++j) p.mates[j] = (char*)d_ws_all[gr * 4 + ((li + 1 + j % 3) & 3)];
    for (int j = 0; j < 8; ++j) p.opps[j] = (char*)d_ws_all[(gr ^ 1) * 4 + (j & 3)];
    for (int j = 0; j < 4; ++j) p.oout[j] = (float*)d_out_all[(gr ^ 1) * 4 + j];
    p.rank = rank;
    hipLaunchKernelGGL(ag_gemm_kernel, dim3(GRID), dim3(BLOCK), LDS_BYTES, stream, p);
    HIP_CHECK(hipGetLastError());
}
```

```cpp
#include <hip/hip_runtime.h>
#include <rccl/rccl.h>
#include <cstdio>
#include <type_traits>

#define HIP_CHECK(expr) do { hipError_t _e = (expr); if (_e != hipSuccess) { \
    fprintf(stderr, "HIP error %s at %s:%d\n", hipGetErrorString(_e), __FILE__, __LINE__); \
    return; } } while (0)

typedef float f4 __attribute__((ext_vector_type(4)));
typedef unsigned u4 __attribute__((ext_vector_type(4)));
typedef _Float16 h8 __attribute__((ext_vector_type(8)));
typedef __attribute__((address_space(1))) unsigned gu32;
typedef __attribute__((address_space(3))) unsigned lu32;

constexpr int N1 = 12;
constexpr int N2 = 16 - N1;
constexpr int NR = 8;
constexpr int KD = 2048;
constexpr int MS = 256;
constexpr int NL = 256;
constexpr unsigned TILE_B = 16 * KD * 2;
constexpr unsigned X_BYTES = 64 * TILE_B;
constexpr unsigned IMG_B = KD * 32 * 2;
constexpr unsigned W_OFF = X_BYTES;
constexpr unsigned PAR = X_BYTES + 8 * IMG_B;
constexpr unsigned NBUF = 3;
constexpr unsigned CTR_OFF = NBUF * PAR;
constexpr unsigned PFLAG_OFF = CTR_OFF + 256 * 64;
constexpr unsigned RFLAG_OFF = PFLAG_OFF + 256;
constexpr unsigned TMO_OFF = RFLAG_OFF + 512;
constexpr unsigned WS_NEED = TMO_OFF + 256;
constexpr unsigned OUT_BYTES = 2048u * NL * 4;
constexpr unsigned SENT = 0xFFFFFFFFu;
constexpr unsigned QNAN = 0x7FC00000u;
constexpr int GRID = 256;
constexpr int BLOCK = 704;
constexpr unsigned PART_B = 8 * 16384;
constexpr int LDS_BYTES = PART_B + 128;
constexpr unsigned DEADLINE = 50000000u;
constexpr float WSCALE = 64.f, WINV = 1.f / 64.f;

#define ITP 58
#define WTP 56
#define IT0 45
#define GT 700
#define FSP 60
#define HKDELAY 100
constexpr int IWIN = 8, WWIN = 8, XWIN = 4, ZWIN = 2;

struct Params {
    const float* x;
    const float* w;
    char* myws;
    int rank;
    int pad;
    float* out;
    char* mates[6];
    char* opps[8];
    float* oout[4];
};

__device__ __forceinline__ unsigned uni(unsigned v) { return __builtin_amdgcn_readfirstlane(v); }
__device__ __forceinline__ unsigned now32() { return (unsigned)__builtin_amdgcn_s_memrealtime(); }
__device__ __forceinline__ unsigned ld_agent(const char* p) {
    return __hip_atomic_load((gu32*)p, __ATOMIC_RELAXED, __HIP_MEMORY_SCOPE_AGENT);
}
__device__ __forceinline__ unsigned ld_sys(const char* p) {
    return __hip_atomic_load((gu32*)p, __ATOMIC_RELAXED, __HIP_MEMORY_SCOPE_SYSTEM);
}
__device__ __forceinline__ void st_agent(char* p, unsigned v) {
    __hip_atomic_store((gu32*)p, v, __ATOMIC_RELAXED, __HIP_MEMORY_SCOPE_AGENT);
}
__device__ __forceinline__ unsigned lds_ld(lu32* a) { return __hip_atomic_load(a, __ATOMIC_RELAXED, __HIP_MEMORY_SCOPE_WORKGROUP); }
__device__ __forceinline__ void lds_st(lu32* a, unsigned v) { __hip_atomic_store(a, v, __ATOMIC_RELAXED, __HIP_MEMORY_SCOPE_WORKGROUP); }
__device__ __forceinline__ __amdgpu_buffer_rsrc_t mkrs(const void* p, unsigned bytes) {
    const unsigned long long a = (unsigned long long)p;
    const unsigned lo = uni((unsigned)a), hi = uni((unsigned)(a >> 32));
    return __builtin_amdgcn_make_buffer_rsrc((void*)(((unsigned long long)hi << 32) | lo), 0, bytes, 0x00020000);
}
__device__ __forceinline__ f4 bld(__amdgpu_buffer_rsrc_t r, unsigned vo, unsigned so) {
    return __builtin_bit_cast(f4, __builtin_amdgcn_raw_buffer_load_b128(r, vo, so, 0));
}
__device__ __forceinline__ u4 cld(__amdgpu_buffer_rsrc_t r, unsigned vo, unsigned so) {
    return __builtin_amdgcn_raw_buffer_load_b128(r, vo, so, 17);
}
__device__ __forceinline__ void cst(u4 v, __amdgpu_buffer_rsrc_t r, unsigned vo, unsigned so) {
    __builtin_amdgcn_raw_buffer_store_b128(v, r, vo, so, 17);
}
__device__ __forceinline__ u4 nz(u4 v) {
    #pragma unroll
    for (int e = 0; e < 4; ++e) v[e] = v[e] == 0u ? 0x80000000u : v[e];
    return v;
}
__device__ __forceinline__ u4 cvt8(f4 a, f4 b) {
    h8 h;
    h[0] = (_Float16)a[0]; h[1] = (_Float16)a[1]; h[2] = (_Float16)a[2]; h[3] = (_Float16)a[3];
    h[4] = (_Float16)b[0]; h[5] = (_Float16)b[1]; h[6] = (_Float16)b[2]; h[7] = (_Float16)b[3];
    return nz(__builtin_bit_cast(u4, h));
}
__device__ __forceinline__ unsigned min4(u4 v) { return min(min(v[0], v[1]), min(v[2], v[3])); }
__device__ __forceinline__ unsigned max4(u4 v) { return max(max(v[0], v[1]), max(v[2], v[3])); }
__device__ __forceinline__ bool all_set(bool b) { return __ballot(b) == ~0ull; }
__device__ __forceinline__ void until(unsigned t0, unsigned ticks) {
    while (now32() - t0 < ticks) __builtin_amdgcn_s_sleep(1);
}
template <int N> __device__ __forceinline__ void vm_wait() { asm volatile("s_waitcnt vmcnt(%0)" :: "i"(N) : "memory"); }

__device__ __forceinline__ float silu(float y) {
    const float c = fminf(fmaxf(y, -60.f), 60.f);
    return y * __builtin_amdgcn_rcpf(1.f + __expf(-c));
}

__device__ __forceinline__ bool expired(char* myws, unsigned call, unsigned t0, unsigned& tries) {
    tries = uni(tries + 1u);
    if ((tries & 31u) != 0u) return false;
    if (uni(ld_agent(myws + TMO_OFF)) == call || now32() - t0 > DEADLINE) {
        st_agent(myws + TMO_OFF, call);
        return true;
    }
    return false;
}

__device__ __forceinline__ void w_to_lds(const f4 (&r)[8], char* lw, int sw) {
    #pragma unroll
    for (int i = 0; i < 4; ++i) {
        h8 h;
        #pragma unroll
        for (int e = 0; e < 8; ++e) h[e] = (_Float16)(r[e][i] * WSCALE);
        *(h8*)(lw + ((i ^ sw) * 16)) = h;
    }
}

__global__ void __launch_bounds__(BLOCK, 3) ag_gemm_kernel(Params p) {
    extern __shared__ __attribute__((aligned(16))) char smem[];
    const int lane = threadIdx.x & 63;
    const int wave = uni(threadIdx.x >> 6);
    const int bid = blockIdx.x;
    const float* px = p.x;
    const float* pw = p.w;
    char* myws = p.myws;
    const int rank = p.rank;
    asm volatile("" :: "s"(px), "s"(pw), "s"(myws), "s"(rank));
    const int gr = rank >> 2, li = rank & 3;
    const unsigned t0 = now32();

    char* ctrp = myws + CTR_OFF + bid * 64 + wave * 4;
    lu32* xdone = (lu32*)(smem + PART_B);
    const int q = lane & 7, kgl = lane >> 3;
    const int lwo = (((kgl >> 2) * 2 + (q >> 2)) * 64 + (kgl & 3) * 16 + 4 * (q & 3)) * 16;
    const int lr0 = ((lane >> 4) * 16 + ((lane & 15) ^ ((lane >> 3) & 1))) * 16;
    const int lr1 = ((lane >> 4) * 16 + ((lane & 15) ^ (((lane >> 3) & 1) | 2))) * 16 + 1024;

    if (__builtin_expect(bid < 32 && wave == 0, 1)) {
        const int j = bid;
        __amdgpu_buffer_rsrc_t rxin = mkrs(px, MS * KD * 4);
        const unsigned xv = (unsigned)(((lane & 15) * KD + (lane >> 4) * 8 + j * 32) * 4);
        constexpr int NU = 2 * N1, PB = 12;
        f4 xa[PB], xb[PB];
        unsigned callv = 0;
        #pragma unroll
        for (int u = 0; u < PB; ++u) {
            xa[u] = bld(rxin, xv, (u >> 1) * 16 * KD * 4 + (u & 1) * 4096); xb[u] = bld(rxin, xv + 16, (u >> 1) * 16 * KD * 4 + (u & 1) * 4096);
            if (u == 0) { __builtin_amdgcn_sched_barrier(0); callv = ld_agent(ctrp); __builtin_amdgcn_sched_barrier(0); }
        }
        __amdgpu_buffer_rsrc_t rs[4];
        const int rot = j % 3;
        #pragma unroll
        for (int d = 0; d < 3; ++d) rs[d] = mkrs(p.mates[rot + d], NBUF * PAR);
        rs[3] = mkrs(myws, NBUF * PAR);
        __builtin_amdgcn_sched_barrier(0);
        const unsigned call = uni(callv) + 1u;
        const unsigned boff = (call % NBUF) * PAR + li * 16 * TILE_B + j * 1024;
        const unsigned voff = lane * 16;
        #pragma unroll
        for (int u = 0; u < NU; ++u) {
            const u4 v = cvt8(xa[u % PB], xb[u % PB]);
            if (u >= 2) until(t0, IT0 + (u - 1) * ITP);
            #pragma unroll
            for (int d = 0; d < 4; ++d) cst(v, rs[d], voff, boff + (u >> 1) * TILE_B + (u & 1) * 32768);
            if (u + PB < NU) {
                const unsigned so = ((u + PB) >> 1) * 16 * KD * 4 + ((u + PB) & 1) * 4096;
                xa[u % PB] = bld(rxin, xv, so); xb[u % PB] = bld(rxin, xv + 16, so);
            }
            if (u >= 1) asm volatile("s_waitcnt vmcnt(%0)" :: "i"(IWIN + (u + PB - 1 < NU ? 2 : 0) + (u + PB < NU ? 2 : 0)) : "memory");
            __builtin_amdgcn_sched_barrier(0);
        }
        st_agent(ctrp, call);
        return;
    }
    if (__builtin_expect(bid < 64 && wave == 0, 1)) {
        const int jj = bid - 32;
        const int dq = jj >> 3, kq = jj & 7;
        __amdgpu_buffer_rsrc_t rwin = mkrs(pw, KD * NL * 4);
        const unsigned wv = (unsigned)(((kq * 64 + kgl * 8) * NL + dq * 64 + q * 4) * 4);
        f4 r[4][8];
        auto wload = [&](int m) {
            #pragma unroll
            for (int e = 0; e < 8; ++e) r[m & 3][e] = bld(rwin, wv + (e & 3) * 1024, (m & 3) * 524288 + (m >> 2) * 128 + (e >> 2) * 4096);
        };
        wload(0);
        __builtin_amdgcn_sched_barrier(0);
        const unsigned callv = ld_agent(ctrp);
        __builtin_amdgcn_sched_barrier(0);
        #pragma unroll
        for (int m = 1; m < 4; ++m) wload(m);
        __amdgpu_buffer_rsrc_t rs[4];
        #pragma unroll
        for (int d = 0; d < 4; ++d) rs[d] = mkrs(p.opps[dq + d], NBUF * PAR);
        __builtin_amdgcn_sched_barrier(0);
        const unsigned call = uni(callv) + 1u;
        const unsigned par = (call % NBUF) * PAR;
        const unsigned woff = par + W_OFF + li * 2 * IMG_B + kq * 4096;
        const unsigned voff = lane * 16;
        auto wpark = [&](int m) { w_to_lds(r[m & 3], smem + m * 4096 + lwo, q >> 1); };
        auto wsend = [&](int m) {
            u4 v[4];
            #pragma unroll
            for (int u = 0; u < 4; ++u) v[u] = nz(*(const u4*)(smem + m * 4096 + (u >> 1) * 2048 + ((u & 1) ? lr1 : lr0)));
            if (m >= 2) until(t0, IT0 + 20 + (m - 1) * WTP);
            #pragma unroll
            for (int u = 0; u < 4; ++u) cst(v[u], rs[0], voff, woff + (m >> 2) * IMG_B + (m & 3) * 32768 + u * 1024);
            __builtin_amdgcn_sched_barrier(0);
        };
        wpark(0); wsend(0);
        wpark(1); wsend(1);
        wpark(2); wpark(3);
        __builtin_amdgcn_sched_barrier(0);
        #pragma unroll
        for (int m = 4; m < 8; ++m) wload(m);
        __builtin_amdgcn_sched_barrier(0);
        wsend(2);
        #pragma unroll
        for (int m = 4; m < 8; ++m) wpark(m);
        __builtin_amdgcn_sched_barrier(0);
        __amdgpu_buffer_rsrc_t rxin = mkrs(px, MS * KD * 4);
        const unsigned xv = (unsigned)(((N1 * 16 + (lane & 15)) * KD + (lane >> 4) * 8 + jj * 32) * 4);
        f4 xa[2 * N2], xb[2 * N2];
        #pragma unroll
        for (int u = 0; u < 2 * N2; ++u) {
            xa[u] = bld(rxin, xv, (u >> 1) * 16 * KD * 4 + (u & 1) * 4096); xb[u] = bld(rxin, xv + 16, (u >> 1) * 16 * KD * 4 + (u & 1) * 4096);
        }
        __builtin_amdgcn_sched_barrier(0);
        wsend(3);
        wsend(4);
        #pragma unroll
        for (int m = 5; m < 8; ++m) { vm_wait<WWIN>(); wsend(m); }
        const unsigned boff = par + (li * 16 + N1) * TILE_B + jj * 1024;
        #pragma unroll
        for (int u = 0; u < 2 * N2; ++u) {
            const u4 v = cvt8(xa[u], xb[u]);
            #pragma unroll
            for (int d = 0; d < 4; ++d) cst(v, rs[d], voff, boff + (u >> 1) * TILE_B + (u & 1) * 32768);
            vm_wait<XWIN>();
            __builtin_amdgcn_sched_barrier(0);
        }
        lds_st(xdone, call);
        st_agent(ctrp, call);
        return;
    }
    if (bid < 64) {
        if (bid < 32 && (wave > 2 || (wave == 2 && bid != 31))) return;
        const unsigned call = uni(ld_agent(ctrp)) + 1u;
        st_agent(ctrp, call);
        if (bid < 32 && wave == 2) {
            unsigned tries = 0;
            while (!all_set(ld_agent(myws + PFLAG_OFF + lane * 4) == call)) {
                if (expired(myws, call, t0, tries)) return;
                __builtin_amdgcn_s_sleep(2);
            }
            if (lane < 4)
                __hip_atomic_store((gu32*)(p.opps[lane] + RFLAG_OFF + li * 128), call, __ATOMIC_RELAXED, __HIP_MEMORY_SCOPE_SYSTEM);
            return;
        }
        if (wave == 1) {
            const int ti = bid & 15;
            __amdgpu_buffer_rsrc_t ro = mkrs(p.out, OUT_BYTES);
            const unsigned so = (unsigned)((((ti < N1 ? gr ^ 1 : gr) * 4 + (bid >> 4)) * MS + ti * 16) * NL * 4);
            const u4 sv = {SENT, SENT, SENT, SENT};
            if (HKDELAY) until(t0, HKDELAY);
            #pragma unroll
            for (int e = 0; e < 16; ++e) cst(sv, ro, (unsigned)(lane * 16), so + e * 1024);
            vm_wait<0>();
            st_agent(myws + PFLAG_OFF + bid * 4, call);
        }
        if (bid < 32) return;
        while (uni(lds_ld(xdone)) != call && now32() - t0 < 5000u) __builtin_amdgcn_s_sleep(8);
        __amdgpu_buffer_rsrc_t rz = mkrs(myws + ((call + 2u) % NBUF) * PAR + (bid - 32) * (PAR / 32), PAR / 32);
        const u4 zero = {0u, 0u, 0u, 0u};
        #pragma unroll
        for (int u = 0; u < 16; ++u) {
            cst(zero, rz, (unsigned)(lane * 16), (unsigned)((u * 10 + wave - 1) * 1024));
            vm_wait<ZWIN>();
        }
        return;
    }

    const int xg = bid & 7;
    const bool foreign = bid < 192;
    const int cidx = foreign ? (bid - 64) >> 4 : 8 + ((bid - 192) >> 3);
    const int fpar = (bid >> 3) & 1;
    const unsigned tmask = foreign ? (0x55u << fpar) : 0xFFu;
    const int mstep = foreign ? 2 : 1;
    const unsigned callv = ld_agent(ctrp);
    lu32* cnt = (lu32*)(smem + PART_B);
    lu32* rdy = cnt + 8;
    lu32* abortp = cnt + 16;
    const int l = xg >> 1, tp = xg & 1;
    const int ncol = foreign ? li * 64 + (cidx & 1) * 32 : (cidx - 8) * 32;
    constexpr int m2 = N1 / 2;
    static_assert(N1 % 2 == 0 && m2 % 2 == 0, "tile streams");
    auto late = [&](unsigned& tries) -> bool {
        tries = uni(tries + 1u);
        if ((tries & 127u) != 0u) return false;
        return uni(lds_ld(abortp)) != 0u || now32() - t0 > DEADLINE + 1000000u;
    };

    if (wave >= 8) {
        if (wave == 8) lds_st(cnt + (lane & 31), 0u);
        asm volatile("s_waitcnt lgkmcnt(0)" ::: "memory");
        __builtin_amdgcn_s_barrier();
        const unsigned call = uni(callv) + 1u;
        st_agent(ctrp, call);
        __builtin_amdgcn_s_setprio(3);
        if (wave == 8) {
            const char* xb = myws + (call % NBUF) * PAR + (l * 16 + tp) * TILE_B + 1020 + (lane & 31) * 1024;
            const bool hi = lane >= 32;
            int ma = foreign ? fpar : 0, mb = foreign ? m2 + fpar : m2;
            unsigned ka = 0, kb = 0, tries = 0;
            while (ma < m2 || mb < 8) {
                const unsigned oa = (unsigned)min(ma, m2 - 1) * (2 * TILE_B), ob = (unsigned)min(mb, 7) * (2 * TILE_B);
                const unsigned long long bal = __ballot(ld_sys(xb + (hi ? ob : oa)) != 0u);
                #pragma unroll
                for (int h = 0; h < 2; ++h) {
                    unsigned b = (unsigned)(bal >> (32 * h));
                    int& mm = h ? mb : ma; unsigned& kk = h ? kb : ka;
                    if (mm < (h ? 8 : m2)) {
                        b &= b >> 1; b &= b >> 2; b &= 0x11111111u;
                        b = (b | (b >> 3)) & 0x03030303u; b = (b | (b >> 6)) & 0x000F000Fu; b = (b | (b >> 12)) & 0xFFu;
                        if (b != kk) { kk = b; lds_st(rdy + mm, b); }
                        if (b == 0xFFu) { mm += mstep; kk = 0; }
                    }
                }
                if (expired(myws, call, t0, tries)) { lds_st(abortp, 1u); break; }
                if (bal == 0ull) __builtin_amdgcn_s_sleep(2);
            }
        } else {
            __amdgpu_buffer_rsrc_t ro = mkrs(foreign ? (const void*)p.oout[cidx >> 1] : (const void*)p.out, OUT_BYTES);
            const int fh = wave - 9;
            const int fro = ((lane >> 2) & 1) * 1024 + ((lane & 3) * 16 + fh * 8 + (lane >> 3)) * 16;
            const unsigned fso = (unsigned)((fh * 8 + (lane >> 3)) * NL * 4 + (lane & 7) * 16);
            unsigned pend = tmask, tries = 0, bad = 0;
            unsigned nxt = t0 + GT + (unsigned)(((bid * 2 + fh) * 37) % 64) * FSP / 64;
            if (foreign) {
                while (uni(ld_sys(myws + RFLAG_OFF + (cidx >> 1) * 128)) != call) {
                    if (expired(myws, call, t0, tries)) { lds_st(abortp, 1u); pend = 0u; break; }
                    __builtin_amdgcn_s_sleep(8);
                }
                tries = 0;
            }
            while (pend) {
                unsigned mk = (unsigned)__ballot(lds_ld(cnt + (lane & 7)) == 8u) & pend;
                if (!mk) {
                    if (late(tries)) { bad = 1; mk = pend; }
                    else { __builtin_amdgcn_s_sleep(1); continue; }
                }
                asm volatile("" ::: "memory");
                const int m = __builtin_ctz(mk);
                pend &= ~(1u << m);
                f4 pa[8];
                #pragma unroll
                for (int v = 0; v < 8; ++v) pa[v] = *(const f4*)(smem + v * 16384 + m * 2048 + fro);
                const f4 a0 = ((pa[0] + pa[1]) + (pa[2] + pa[3])) + ((pa[4] + pa[5]) + (pa[6] + pa[7]));
                u4 o0;
                #pragma unroll
                for (int r = 0; r < 4; ++r) {
                    o0[r] = __builtin_bit_cast(unsigned, silu(a0[r] * WINV));
                    o0[r] = (bad || o0[r] == SENT) ? QNAN : o0[r];
                }
                const int ti = 2 * m + tp;
                const int srcr = ((ti < N1) ? gr : (gr ^ 1)) * 4 + l;
                if (foreign) {
                    while ((int)(now32() - nxt) < 0) __builtin_amdgcn_s_sleep(1);
                    nxt = now32() + FSP;
                }
                cst(o0, ro, fso, (unsigned)(((srcr * MS + ti * 16) * NL + ncol) * 4));
                if (bad) tries = 127u;
            }
        }
        return;
    }

    char* myreg = smem + wave * 16384;
    h8 w0[8], w1[8];
    unsigned call;
    bool bad = false;
    if (!foreign) {
        __amdgpu_buffer_rsrc_t rwin = mkrs(pw, KD * NL * 4);
        const unsigned wv = (unsigned)(((wave * 128 + kgl * 8) * NL + ncol + q * 4) * 4);
        f4 r[4][8];
        #pragma unroll
        for (int b = 0; b < 4; ++b) {
            #pragma unroll
            for (int j = 0; j < 8; ++j) r[b][j] = bld(rwin, wv + (j & 3) * 1024, ((b >> 1) * 1024 + (b & 1) * 64) * NL * 4 + (j >> 2) * 4096);
        }
        __builtin_amdgcn_s_barrier();
        #pragma unroll
        for (int b = 0; b < 4; ++b) w_to_lds(r[b], myreg + b * 4096 + lwo, q >> 1);
        #pragma unroll
        for (int s = 0; s < 8; ++s) { w0[s] = *(const h8*)(myreg + s * 2048 + lr0); w1[s] = *(const h8*)(myreg + s * 2048 + lr1); }
        call = uni(callv) + 1u;
    } else {
        __builtin_amdgcn_s_barrier();
        call = uni(callv) + 1u;
        const char* ib = myws + (call % NBUF) * PAR + W_OFF + cidx * IMG_B + wave * 8192;
        __amdgpu_buffer_rsrc_t ri = mkrs(ib, 65536 + 8192);
        const char* pp = ib + (lane & 2) * 32768 + (lane & 1) * 4096 + 4092;
        unsigned tries = 0;
        bool have;
        do {
            while (!all_set(ld_sys(pp) != 0u)) {
                if (expired(myws, call, t0, tries)) { bad = true; break; }
                __builtin_amdgcn_s_sleep(4);
            }
            unsigned m = 0xffffffffu;
            #pragma unroll
            for (int s = 0; s < 8; ++s) {
                const u4 a = cld(ri, (unsigned)(lane * 16 + (s & 1) * 2048), (s >> 2) * 65536 + ((s >> 1) & 1) * 4096);
                const u4 b = cld(ri, (unsigned)(lane * 16 + (s & 1) * 2048 + 1024), (s >> 2) * 65536 + ((s >> 1) & 1) * 4096);
                w0[s] = __builtin_bit_cast(h8, a); w1[s] = __builtin_bit_cast(h8, b);
                m = min(m, min(min4(a), min4(b)));
            }
            have = all_set(m != 0u);
            if (!have && expired(myws, call, t0, tries)) bad = true;
        } while (!have && !bad);
        if (bad) lds_st(abortp, 1u);
    }
    st_agent(ctrp, call);

    __amdgpu_buffer_rsrc_t rx = mkrs(myws + (call % NBUF) * PAR + (l * 16 + tp) * TILE_B + wave * 4096, 15 * TILE_B);

    auto process = [&](auto nt_tag, int ma, int mb) -> unsigned {
        constexpr int NT = decltype(nt_tag)::value;
        u4 a[NT * 8];
        int so[2] = {(int)uni(ma * (2 * TILE_B)), (int)uni(mb * (2 * TILE_B))};
        asm volatile("" : "+s"(so[0]), "+s"(so[1]));
        #pragma unroll
        for (int t = 0; t < NT; ++t) {
            #pragma unroll
            for (int s = 0; s < 8; ++s) {
                a[t * 8 + s] = cld(rx, (unsigned)(lane * 16 + (s & 3) * 1024), so[t] + (s >> 2) * 32768);
                if ((s & 3) == 3) __builtin_amdgcn_sched_barrier(0);
            }
        }
        unsigned okm = 0;
        #pragma unroll
        for (int t = 0; t < NT; ++t) {
            f4 n0 = {0.f, 0.f, 0.f, 0.f}, n1 = {0.f, 0.f, 0.f, 0.f};
            unsigned mnA = 0xffffffffu, mnB = 0xffffffffu;
            #pragma unroll
            for (int s = 0; s < 4; ++s) {
                const h8 av = __builtin_bit_cast(h8, a[t * 8 + s]);
                n0 = __builtin_amdgcn_mfma_f32_16x16x32_f16(w0[s], av, n0, 0, 0, 0);
                n1 = __builtin_amdgcn_mfma_f32_16x16x32_f16(w1[s], av, n1, 0, 0, 0);
                mnA = min(mnA, min4(a[t * 8 + s]));
            }
            #pragma unroll
            for (int s = 4; s < 8; ++s) mnB = min(mnB, min4(a[t * 8 + s]));
            const bool okB = all_set(mnB != 0u);
            if (okB) {
                #pragma unroll
                for (int s = 4; s < 8; ++s) {
                    const h8 av = __builtin_bit_cast(h8, a[t * 8 + s]);
                    n0 = __builtin_amdgcn_mfma_f32_16x16x32_f16(w0[s], av, n0, 0, 0, 0);
                    n1 = __builtin_amdgcn_mfma_f32_16x16x32_f16(w1[s], av, n1, 0, 0, 0);
                }
            }
            const int m = t ? mb : ma;
            *(f4*)(myreg + m * 2048 + lane * 16) = n0;
            *(f4*)(myreg + m * 2048 + 1024 + lane * 16) = n1;
            if (all_set(mnA != 0u)) okm |= (okB ? 1u : 256u) << m;
            __builtin_amdgcn_sched_barrier(0);
        }
        return okm;
    };
    auto processB = [&](int m) -> unsigned {
        u4 a[4];
        int so = (int)uni(m * (2 * TILE_B) + 32768);
        asm volatile("" : "+s"(so));
        #pragma unroll
        for (int s = 0; s < 4; ++s) a[s] = cld(rx, (unsigned)(lane * 16 + s * 1024), so);
        f4 n0 = *(const f4*)(myreg + m * 2048 + lane * 16), n1 = *(const f4*)(myreg + m * 2048 + 1024 + lane * 16);
        __builtin_amdgcn_sched_barrier(0);
        unsigned mn = 0xffffffffu;
        #pragma unroll
        for (int s = 0; s < 4; ++s) {
            const h8 av = __builtin_bit_cast(h8, a[s]);
            n0 = __builtin_amdgcn_mfma_f32_16x16x32_f16(w0[4 + s], av, n0, 0, 0, 0);
            n1 = __builtin_amdgcn_mfma_f32_16x16x32_f16(w1[4 + s], av, n1, 0, 0, 0);
            mn = min(mn, min4(a[s]));
        }
        if (!all_set(mn != 0u)) return 0u;
        *(f4*)(myreg + m * 2048 + lane * 16) = n0;
        *(f4*)(myreg + m * 2048 + 1024 + lane * 16) = n1;
        return 1u << m;
    };

    {
        unsigned pend = bad ? 0u : tmask, pendB = 0u, tries = 0;
        bool lastB = false;
        while (pend | pendB) {
            unsigned okm = 0;
            if (late(tries)) { bad = true; break; }
            unsigned mk = (unsigned)__ballot((lds_ld(rdy + (lane & 7)) >> wave) & 1u) & pend;
            if (pendB && !lastB) mk = 0;
            lastB = mk == 0u;
            if (mk) {
                const int ma = __builtin_ctz(mk); mk &= mk - 1;
                __builtin_amdgcn_s_setprio(2);
                if (mk) okm = process(std::integral_constant<int, 2>{}, ma, __builtin_ctz(mk));
                else okm = process(std::integral_constant<int, 1>{}, ma, ma);
                pend &= ~(okm | (okm >> 8));
                pendB |= okm >> 8;
                okm &= 0xFFu;
            } else if (pendB) {
                __builtin_amdgcn_s_setprio(2);
                unsigned t = pendB;
                do { const int m = __builtin_ctz(t); t &= t - 1; okm |= processB(m); } while (t);
                pendB &= ~okm;
            } else { __builtin_amdgcn_s_sleep(1); continue; }
            asm volatile("s_waitcnt lgkmcnt(0)" ::: "memory");
            if (lane < 8 && ((okm >> lane) & 1u)) __hip_atomic_fetch_add(cnt + lane, 1u, __ATOMIC_RELAXED, __HIP_MEMORY_SCOPE_WORKGROUP);
            __builtin_amdgcn_s_setprio(0);
        }
    }
    if (!bad && !foreign) {
        const int ti = 2 * wave + tp;
        const int srcr = ((ti < N1) ? gr ^ 1 : gr) * 4 + l;
        __amdgpu_buffer_rsrc_t ro = mkrs(p.out, OUT_BYTES);
        const unsigned om = (unsigned)(((lane >> 3) * NL) * 4 + (lane & 7) * 16);
        const unsigned so = (unsigned)(((srcr * MS + ti * 16) * NL + ncol) * 4);
        unsigned tries = 0;
        u4 a0 = cld(ro, om, so), b0 = cld(ro, om, so + 8 * NL * 4);
        __builtin_amdgcn_s_sleep(5);
        u4 a1 = cld(ro, om, so), b1 = cld(ro, om, so + 8 * NL * 4);
        __builtin_amdgcn_s_sleep(5);
        for (;;) {
            u4 a2 = cld(ro, om, so), b2 = cld(ro, om, so + 8 * NL * 4);
            __builtin_amdgcn_sched_barrier(0);
            if (all_set(max(max4(a0), max4(b0)) != SENT)) break;
            a0 = cld(ro, om, so); b0 = cld(ro, om, so + 8 * NL * 4);
            __builtin_amdgcn_sched_barrier(0);
            if (all_set(max(max4(a1), max4(b1)) != SENT)) break;
            a1 = cld(ro, om, so); b1 = cld(ro, om, so + 8 * NL * 4);
            __builtin_amdgcn_sched_barrier(0);
            if (all_set(max(max4(a2), max4(b2)) != SENT)) break;
            if (expired(myws, call, t0, tries)) break;
        }
    }
}

extern "C" void kernel_launch(int rank, int nranks, ncclComm_t comm,
                              hipStream_t stream,
                              void* const* d_in, const int* in_sizes, int n_in,
                              void* d_out, int out_size,
                              const int* in_dtypes, int out_dtype,
                              void* const* d_out_all,
                              void* const* d_ws_all, size_t ws_bytes) {
    if (nranks != NR || ws_bytes < WS_NEED || n_in < 2 || rank < 0 || rank >= NR ||
        in_sizes[0] != MS * KD || in_sizes[1] != KD * NL || out_size != NR * MS * NL) {
        fprintf(stderr, "unexpected configuration\n");
        return;
    }
    static bool init = false;
    if (!init) {
        HIP_CHECK(hipFuncSetAttribute((const void*)ag_gemm_kernel, hipFuncAttributeMaxDynamicSharedMemorySize, LDS_BYTES));
        init = true;
    }
    Params p;
    const int gr = rank >> 2, li = rank & 3;
    p.x = (const float*)d_in[0];
    p.w = (const float*)d_in[1];
    p.myws = (char*)d_ws_all[rank];
    p.rank = rank;
    p.pad = 0;
    p.out = (float*)d_out;
    for (int j = 0; j < 6; ++j) p.mates[j] = (char*)d_ws_all[gr * 4 + ((li + 1 + j % 3) & 3)];
    for (int j = 0; j < 8; ++j) p.opps[j] = (char*)d_ws_all[(gr ^ 1) * 4 + (j & 3)];
    for (int j = 0; j < 4; ++j) p.oout[j] = (float*)d_out_all[(gr ^ 1) * 4 + j];
    hipLaunchKernelGGL(ag_gemm_kernel, dim3(GRID), dim3(BLOCK), LDS_BYTES, stream, p);
    HIP_CHECK(hipGetLastError());
}
```

```cpp
#include <hip/hip_runtime.h>
#include <rccl/rccl.h>
#include <cstdio>
#include <type_traits>

#define HIP_CHECK(expr) do { hipError_t _e = (expr); if (_e != hipSuccess) { \
    fprintf(stderr, "HIP error %s at %s:%d\n", hipGetErrorString(_e), __FILE__, __LINE__); \
    return; } } while (0)

typedef float f4 __attribute__((ext_vector_type(4)));
typedef unsigned u4 __attribute__((ext_vector_type(4)));
typedef _Float16 h8 __attribute__((ext_vector_type(8)));
typedef __attribute__((address_space(1))) unsigned gu32;
typedef __attribute__((address_space(3))) unsigned lu32;

constexpr int N1 = 12;
constexpr int N2 = 16 - N1;
constexpr int NR = 8;
constexpr int KD = 2048;
constexpr int MS = 256;
constexpr int NL = 256;
constexpr unsigned TILE_B = 16 * KD * 2;
constexpr unsigned X_BYTES = 64 * TILE_B;
constexpr unsigned IMG_B = KD * 32 * 2;
constexpr unsigned W_OFF = X_BYTES;
constexpr unsigned PAR = X_BYTES + 8 * IMG_B;
constexpr unsigned NBUF = 3;
constexpr unsigned CTR_OFF = NBUF * PAR;
constexpr unsigned PFLAG_OFF = CTR_OFF + 256 * 64;
constexpr unsigned RFLAG_OFF = PFLAG_OFF + 256;
constexpr unsigned TMO_OFF = RFLAG_OFF + 512;
constexpr unsigned WS_NEED = TMO_OFF + 256;
constexpr unsigned OUT_BYTES = 2048u * NL * 4;
constexpr unsigned SENT = 0xFFFFFFFFu;
constexpr unsigned QNAN = 0x7FC00000u;
constexpr int GRID = 256;
constexpr int BLOCK = 768;
constexpr unsigned PART_B = 8 * 16384;
constexpr int LDS_BYTES = PART_B + 128;
constexpr unsigned DEADLINE = 50000000u;
constexpr float WSCALE = 64.f, WINV = 1.f / 64.f;

constexpr unsigned IT0 = 45;
constexpr unsigned ITP = 58;
constexpr unsigned WTP = 56;
constexpr unsigned GT = 700;
constexpr unsigned FSP = 60;
constexpr unsigned HKT = 100;
constexpr unsigned ZT0 = 150, ZTP = 25;
constexpr int IWIN = 8, WWIN = 8, XWIN = 4;

struct Params {
    const float* x;
    const float* w;
    char* myws;
    int rank;
    int pad;
    float* out;
    char* mates[6];
    char* opps[8];
    float* oout[4];
};

__device__ __forceinline__ unsigned uni(unsigned v) { return __builtin_amdgcn_readfirstlane(v); }
__device__ __forceinline__ unsigned now32() { return (unsigned)__builtin_amdgcn_s_memrealtime(); }
__device__ __forceinline__ unsigned ld_agent(const char* p) {
    return __hip_atomic_load((gu32*)p, __ATOMIC_RELAXED, __HIP_MEMORY_SCOPE_AGENT);
}
__device__ __forceinline__ unsigned ld_sys(const char* p) {
    return __hip_atomic_load((gu32*)p, __ATOMIC_RELAXED, __HIP_MEMORY_SCOPE_SYSTEM);
}
__device__ __forceinline__ void st_agent(char* p, unsigned v) {
    __hip_atomic_store((gu32*)p, v, __ATOMIC_RELAXED, __HIP_MEMORY_SCOPE_AGENT);
}
__device__ __forceinline__ unsigned lds_ld(lu32* a) { return __hip_atomic_load(a, __ATOMIC_RELAXED, __HIP_MEMORY_SCOPE_WORKGROUP); }
__device__ __forceinline__ void lds_st(lu32* a, unsigned v) { __hip_atomic_store(a, v, __ATOMIC_RELAXED, __HIP_MEMORY_SCOPE_WORKGROUP); }
__device__ __forceinline__ __amdgpu_buffer_rsrc_t mkrs(const void* p, unsigned bytes) {
    const unsigned long long a = (unsigned long long)p;
    const unsigned lo = uni((unsigned)a), hi = uni((unsigned)(a >> 32));
    return __builtin_amdgcn_make_buffer_rsrc((void*)(((unsigned long long)hi << 32) | lo), 0, bytes, 0x00020000);
}
__device__ __forceinline__ f4 bld(__amdgpu_buffer_rsrc_t r, unsigned vo, unsigned so) {
    return __builtin_bit_cast(f4, __builtin_amdgcn_raw_buffer_load_b128(r, vo, so, 0));
}
__device__ __forceinline__ u4 cld(__amdgpu_buffer_rsrc_t r, unsigned vo, unsigned so) {
    return __builtin_amdgcn_raw_buffer_load_b128(r, vo, so, 17);
}
__device__ __forceinline__ void cst(u4 v, __amdgpu_buffer_rsrc_t r, unsigned vo, unsigned so) {
    __builtin_amdgcn_raw_buffer_store_b128(v, r, vo, so, 17);
}
__device__ __forceinline__ u4 nz(u4 v) {
    #pragma unroll
    for (int e = 0; e < 4; ++e) v[e] = v[e] == 0u ? 0x80000000u : v[e];
    return v;
}
__device__ __forceinline__ u4 cvt8(f4 a, f4 b) {
    h8 h;
    h[0] = (_Float16)a[0]; h[1] = (_Float16)a[1]; h[2] = (_Float16)a[2]; h[3] = (_Float16)a[3];
    h[4] = (_Float16)b[0]; h[5] = (_Float16)b[1]; h[6] = (_Float16)b[2]; h[7] = (_Float16)b[3];
    return nz(__builtin_bit_cast(u4, h));
}
__device__ __forceinline__ unsigned min4(u4 v) { return min(min(v[0], v[1]), min(v[2], v[3])); }
__device__ __forceinline__ unsigned max4(u4 v) { return max(max(v[0], v[1]), max(v[2], v[3])); }
__device__ __forceinline__ bool all_set(bool b) { return __ballot(b) == ~0ull; }
__device__ __forceinline__ void until(unsigned t0, unsigned ticks) {
    while (now32() - t0 < ticks) __builtin_amdgcn_s_sleep(1);
}
template <int N> __device__ __forceinline__ void vm_wait() { asm volatile("s_waitcnt vmcnt(%0)" :: "i"(N) : "memory"); }

__device__ __forceinline__ float silu(float y) {
    const float c = fminf(fmaxf(y, -60.f), 60.f);
    return y * __builtin_amdgcn_rcpf(1.f + __expf(-c));
}

__device__ __forceinline__ bool expired(char* myws, unsigned call, unsigned t0, unsigned& tries) {
    tries = uni(tries + 1u);
    if ((tries & 31u) != 0u) return false;
    if (uni(ld_agent(myws + TMO_OFF)) == call || now32() - t0 > DEADLINE) {
        st_agent(myws + TMO_OFF, call);
        return true;
    }
    return false;
}

__device__ __forceinline__ void w_to_lds(const f4 (&r)[8], char* lw, int sw) {
    #pragma unroll
    for (int i = 0; i < 4; ++i) {
        h8 h;
        #pragma unroll
        for (int e = 0; e < 8; ++e) h[e] = (_Float16)(r[e][i] * WSCALE);
        *(h8*)(lw + ((i ^ sw) * 16)) = h;
    }
}

__global__ void __launch_bounds__(BLOCK, 3) ag_gemm_kernel(Params p) {
    extern __shared__ __attribute__((aligned(16))) char smem[];
    const int lane = threadIdx.x & 63;
    const int wave = uni(threadIdx.x >> 6);
    const int bid = blockIdx.x;
    const float* px = p.x;
    const float* pw = p.w;
    char* myws = p.myws;
    const int rank = p.rank;
    asm volatile("" :: "s"(px), "s"(pw), "s"(myws), "s"(rank));
    const int gr = rank >> 2, li = rank & 3;
    const unsigned t0 = now32();

    char* ctrp = myws + CTR_OFF + bid * 64 + wave * 4;
    const int q = lane & 7, kgl = lane >> 3;
    const int lwo = (((kgl >> 2) * 2 + (q >> 2)) * 64 + (kgl & 3) * 16 + 4 * (q & 3)) * 16;
    const int lr0 = ((lane >> 4) * 16 + ((lane & 15) ^ ((lane >> 3) & 1))) * 16;
    const int lr1 = ((lane >> 4) * 16 + ((lane & 15) ^ (((lane >> 3) & 1) | 2))) * 16 + 1024;

    if (__builtin_expect(bid < 32 && wave == 0, 1)) {
        const int j = bid;
        __amdgpu_buffer_rsrc_t rxin = mkrs(px, MS * KD * 4);
        const unsigned xv = (unsigned)(((lane & 15) * KD + (lane >> 4) * 8 + j * 32) * 4);
        constexpr int NU = 2 * N1, PB = 12;
        f4 xa[PB], xb[PB];
        unsigned callv = 0;
        #pragma unroll
        for (int u = 0; u < PB; ++u) {
            xa[u] = bld(rxin, xv, (u >> 1) * 16 * KD * 4 + (u & 1) * 4096); xb[u] = bld(rxin, xv + 16, (u >> 1) * 16 * KD * 4 + (u & 1) * 4096);
            if (u == 0) { __builtin_amdgcn_sched_barrier(0); callv = ld_agent(ctrp); __builtin_amdgcn_sched_barrier(0); }
        }
        __amdgpu_buffer_rsrc_t rs[4];
        const int rot = j % 3;
        #pragma unroll
        for (int d = 0; d < 3; ++d) rs[d] = mkrs(p.mates[rot + d], NBUF * PAR);
        rs[3] = mkrs(myws, NBUF * PAR);
        __builtin_amdgcn_sched_barrier(0);
        const unsigned call = uni(callv) + 1u;
        const unsigned boff = (call % NBUF) * PAR + li * 16 * TILE_B + j * 1024;
        const unsigned voff = lane * 16;
        #pragma unroll
        for (int u = 0; u < NU; ++u) {
            const u4 v = cvt8(xa[u % PB], xb[u % PB]);
            if (u >= 2) until(t0, IT0 + (u - 1) * ITP);
            #pragma unroll
            for (int d = 0; d < 4; ++d) cst(v, rs[d], voff, boff + (u >> 1) * TILE_B + (u & 1) * 32768);
            if (u + PB < NU) {
                const unsigned so = ((u + PB) >> 1) * 16 * KD * 4 + ((u + PB) & 1) * 4096;
                xa[u % PB] = bld(rxin, xv, so); xb[u % PB] = bld(rxin, xv + 16, so);
            }
            if (u >= 1) asm volatile("s_waitcnt vmcnt(%0)" :: "i"(IWIN + (u + PB - 1 < NU ? 2 : 0) + (u + PB < NU ? 2 : 0)) : "memory");
            __builtin_amdgcn_sched_barrier(0);
        }
        st_agent(ctrp, call);
        return;
    }
    if (__builtin_expect(bid < 64 && wave == 0, 1)) {
        const int jj = bid - 32;
        const int dq = jj >> 3, kq = jj & 7;
        __amdgpu_buffer_rsrc_t rwin = mkrs(pw, KD * NL * 4);
        const unsigned wv = (unsigned)(((kq * 64 + kgl * 8) * NL + dq * 64 + q * 4) * 4);
        f4 r[4][8];
        auto wload = [&](int m) {
            #pragma unroll
            for (int e = 0; e < 8; ++e) r[m & 3][e] = bld(rwin, wv + (e & 3) * 1024, (m & 3) * 524288 + (m >> 2) * 128 + (e >> 2) * 4096);
        };
        wload(0);
        __builtin_amdgcn_sched_barrier(0);
        const unsigned callv = ld_agent(ctrp);
        __builtin_amdgcn_sched_barrier(0);
        #pragma unroll
        for (int m = 1; m < 4; ++m) wload(m);
        __amdgpu_buffer_rsrc_t rs[4];
        #pragma unroll
        for (int d = 0; d < 4; ++d) rs[d] = mkrs(p.opps[dq + d], NBUF * PAR);
        __builtin_amdgcn_sched_barrier(0);
        const unsigned call = uni(callv) + 1u;
        const unsigned par = (call % NBUF) * PAR;
        const unsigned woff = par + W_OFF + li * 2 * IMG_B + kq * 4096;
        const unsigned voff = lane * 16;
        auto wpark = [&](int m) { w_to_lds(r[m & 3], smem + m * 4096 + lwo, q >> 1); };
        auto wsend = [&](int m) {
            u4 v[4];
            #pragma unroll
            for (int u = 0; u < 4; ++u) v[u] = nz(*(const u4*)(smem + m * 4096 + (u >> 1) * 2048 + ((u & 1) ? lr1 : lr0)));
            if (m >= 2) until(t0, IT0 + 20 + (m - 1) * WTP);
            #pragma unroll
            for (int u = 0; u < 4; ++u) cst(v[u], rs[0], voff, woff + (m >> 2) * IMG_B + (m & 3) * 32768 + u * 1024);
            __builtin_amdgcn_sched_barrier(0);
        };
        wpark(0); wsend(0);
        wpark(1); wsend(1);
        wpark(2); wpark(3);
        __builtin_amdgcn_sched_barrier(0);
        #pragma unroll
        for (int m = 4; m < 8; ++m) wload(m);
        __builtin_amdgcn_sched_barrier(0);
        wsend(2);
        #pragma unroll
        for (int m = 4; m < 8; ++m) wpark(m);
        __builtin_amdgcn_sched_barrier(0);
        __amdgpu_buffer_rsrc_t rxin = mkrs(px, MS * KD * 4);
        const unsigned xv = (unsigned)(((N1 * 16 + (lane & 15)) * KD + (lane >> 4) * 8 + jj * 32) * 4);
        f4 xa[2 * N2], xb[2 * N2];
        #pragma unroll
        for (int u = 0; u < 2 * N2; ++u) {
            xa[u] = bld(rxin, xv, (u >> 1) * 16 * KD * 4 + (u & 1) * 4096); xb[u] = bld(rxin, xv + 16, (u >> 1) * 16 * KD * 4 + (u & 1) * 4096);
        }
        __builtin_amdgcn_sched_barrier(0);
        wsend(3);
        wsend(4);
        #pragma unroll
        for (int m = 5; m < 8; ++m) { vm_wait<WWIN>(); wsend(m); }
        const unsigned boff = par + (li * 16 + N1) * TILE_B + jj * 1024;
        #pragma unroll
        for (int u = 0; u < 2 * N2; ++u) {
            const u4 v = cvt8(xa[u], xb[u]);
            #pragma unroll
            for (int d = 0; d < 4; ++d) cst(v, rs[d], voff, boff + (u >> 1) * TILE_B + (u & 1) * 32768);
            vm_wait<XWIN>();
            __builtin_amdgcn_sched_barrier(0);
        }
        st_agent(ctrp, call);
        return;
    }
    if (bid < 64) {
        if (wave > 2 || (wave == 2 && bid != 31)) return;
        const unsigned call = uni(ld_agent(ctrp)) + 1u;
        st_agent(ctrp, call);
        if (wave == 2) {
            unsigned tries = 0;
            while (!all_set(ld_agent(myws + PFLAG_OFF + lane * 4) == call)) {
                if (expired(myws, call, t0, tries)) return;
                __builtin_amdgcn_s_sleep(2);
            }
            if (lane < 4)
                __hip_atomic_store((gu32*)(p.opps[lane] + RFLAG_OFF + li * 128), call, __ATOMIC_RELAXED, __HIP_MEMORY_SCOPE_SYSTEM);
            return;
        }
        if (wave == 1) {
            const int ti = bid & 15;
            __amdgpu_buffer_rsrc_t ro = mkrs(p.out, OUT_BYTES);
            const unsigned so = (unsigned)((((ti < N1 ? gr ^ 1 : gr) * 4 + (bid >> 4)) * MS + ti * 16) * NL * 4);
            const u4 sv = {SENT, SENT, SENT, SENT};
            until(t0, HKT);
            #pragma unroll
            for (int e = 0; e < 16; ++e) cst(sv, ro, (unsigned)(lane * 16), so + e * 1024);
            vm_wait<0>();
            st_agent(myws + PFLAG_OFF + bid * 4, call);
        }
        return;
    }

    const int xg = bid & 7;
    const bool foreign = bid < 192;
    const int cidx = foreign ? (bid - 64) >> 4 : 8 + ((bid - 192) >> 3);
    const int fpar = (bid >> 3) & 1;
    const unsigned tmask = foreign ? (0x55u << fpar) : 0xFFu;
    const int mstep = foreign ? 2 : 1;
    const unsigned callv = ld_agent(ctrp);
    lu32* cnt = (lu32*)(smem + PART_B);
    lu32* rdy = cnt + 8;
    lu32* abortp = cnt + 16;
    const int l = xg >> 1, tp = xg & 1;
    const int ncol = foreign ? li * 64 + (cidx & 1) * 32 : (cidx - 8) * 32;
    constexpr int m2 = N1 / 2;
    static_assert(N1 % 2 == 0 && m2 % 2 == 0, "tile streams");
    auto late = [&](unsigned& tries) -> bool {
        tries = uni(tries + 1u);
        if ((tries & 127u) != 0u) return false;
        return uni(lds_ld(abortp)) != 0u || now32() - t0 > DEADLINE + 1000000u;
    };

    if (wave == 11) {
        __builtin_amdgcn_s_barrier();
        if (!foreign) return;
        const unsigned call = uni(callv) + 1u;
        st_agent(ctrp, call);
        constexpr unsigned ZB = PAR / 128;
        __amdgpu_buffer_rsrc_t rz = mkrs(myws + ((call + 2u) % NBUF) * PAR + (bid - 64) * ZB, ZB);
        const u4 zero = {0u, 0u, 0u, 0u};
        const unsigned ph = (unsigned)((bid * 29) % 32) * ZTP / 32;
        #pragma unroll 1
        for (unsigned u = 0; u < ZB / 1024; ++u) {
            until(t0, ZT0 + ph + u * ZTP);
            cst(zero, rz, (unsigned)(lane * 16), u * 1024);
        }
        return;
    }
    if (wave >= 8) {
        if (wave == 8) lds_st(cnt + (lane & 31), 0u);
        asm volatile("s_waitcnt lgkmcnt(0)" ::: "memory");
        __builtin_amdgcn_s_barrier();
        const unsigned call = uni(callv) + 1u;
        st_agent(ctrp, call);
        __builtin_amdgcn_s_setprio(3);
        if (wave == 8) {
            const char* xb = myws + (call % NBUF) * PAR + (l * 16 + tp) * TILE_B + 1020 + (lane & 31) * 1024;
            const bool hi = lane >= 32;
            int ma = foreign ? fpar : 0, mb = foreign ? m2 + fpar : m2;
            unsigned ka = 0, kb = 0, tries = 0;
            while (ma < m2 || mb < 8) {
                const unsigned oa = (unsigned)min(ma, m2 - 1) * (2 * TILE_B), ob = (unsigned)min(mb, 7) * (2 * TILE_B);
                const unsigned long long bal = __ballot(ld_sys(xb + (hi ? ob : oa)) != 0u);
                #pragma unroll
                for (int h = 0; h < 2; ++h) {
                    unsigned b = (unsigned)(bal >> (32 * h));
                    int& mm = h ? mb : ma; unsigned& kk = h ? kb : ka;
                    if (mm < (h ? 8 : m2)) {
                        b &= b >> 1; b &= b >> 2; b &= 0x11111111u;
                        b = (b | (b >> 3)) & 0x03030303u; b = (b | (b >> 6)) & 0x000F000Fu; b = (b | (b >> 12)) & 0xFFu;
                        if (b != kk) { kk = b; lds_st(rdy + mm, b); }
                        if (b == 0xFFu) { mm += mstep; kk = 0; }
                    }
                }
                if (expired(myws, call, t0, tries)) { lds_st(abortp, 1u); break; }
                if (bal == 0ull) __builtin_amdgcn_s_sleep(2);
            }
        } else {
            __amdgpu_buffer_rsrc_t ro = mkrs(foreign ? (const void*)p.oout[cidx >> 1] : (const void*)p.out, OUT_BYTES);
            const int fh = wave - 9;
            const int fro = ((lane >> 2) & 1) * 1024 + ((lane & 3) * 16 + fh * 8 + (lane >> 3)) * 16;
            const unsigned fso = (unsigned)((fh * 8 + (lane >> 3)) * NL * 4 + (lane & 7) * 16);
            unsigned pend = tmask, tries = 0, bad = 0;
            unsigned nxt = t0 + GT + (unsigned)(((bid * 2 + fh) * 37) % 64) * FSP / 64;
            if (foreign) {
                while (uni(ld_sys(myws + RFLAG_OFF + (cidx >> 1) * 128)) != call) {
                    if (expired(myws, call, t0, tries)) { lds_st(abortp, 1u); pend = 0u; break; }
                    __builtin_amdgcn_s_sleep(8);
                }
                tries = 0;
            }
            while (pend) {
                unsigned mk = (unsigned)__ballot(lds_ld(cnt + (lane & 7)) == 8u) & pend;
                if (!mk) {
                    if (late(tries)) { bad = 1; mk = pend; }
                    else { __builtin_amdgcn_s_sleep(1); continue; }
                }
                asm volatile("" ::: "memory");
                const int m = __builtin_ctz(mk);
                pend &= ~(1u << m);
                f4 pa[8];
                #pragma unroll
                for (int v = 0; v < 8; ++v) pa[v] = *(const f4*)(smem + v * 16384 + m * 2048 + fro);
                const f4 a0 = ((pa[0] + pa[1]) + (pa[2] + pa[3])) + ((pa[4] + pa[5]) + (pa[6] + pa[7]));
                u4 o0;
                #pragma unroll
                for (int r = 0; r < 4; ++r) {
                    o0[r] = __builtin_bit_cast(unsigned, silu(a0[r] * WINV));
                    o0[r] = (bad || o0[r] == SENT) ? QNAN : o0[r];
                }
                const int ti = 2 * m + tp;
                const int srcr = ((ti < N1) ? gr : (gr ^ 1)) * 4 + l;
                if (foreign) {
                    if (!pend) nxt = t0 + GT;
                    while ((int)(now32() - nxt) < 0) __builtin_amdgcn_s_sleep(1);
                    nxt = now32() + FSP;
                }
                cst(o0, ro, fso, (unsigned)(((srcr * MS + ti * 16) * NL + ncol) * 4));
                if (bad) tries = 127u;
            }
        }
        return;
    }

    char* myreg = smem + wave * 16384;
    h8 w0[8], w1[8];
    unsigned call;
    bool bad = false;
    if (!foreign) {
        __amdgpu_buffer_rsrc_t rwin = mkrs(pw, KD * NL * 4);
        const unsigned wv = (unsigned)(((wave * 128 + kgl * 8) * NL + ncol + q * 4) * 4);
        f4 r[4][8];
        #pragma unroll
        for (int b = 0; b < 4; ++b) {
            #pragma unroll
            for (int j = 0; j < 8; ++j) r[b][j] = bld(rwin, wv + (j & 3) * 1024, ((b >> 1) * 1024 + (b & 1) * 64) * NL * 4 + (j >> 2) * 4096);
        }
        __builtin_amdgcn_s_barrier();
        #pragma unroll
        for (int b = 0; b < 4; ++b) w_to_lds(r[b], myreg + b * 4096 + lwo, q >> 1);
        #pragma unroll
        for (int s = 0; s < 8; ++s) { w0[s] = *(const h8*)(myreg + s * 2048 + lr0); w1[s] = *(const h8*)(myreg + s * 2048 + lr1); }
        call = uni(callv) + 1u;
    } else {
        __builtin_amdgcn_s_barrier();
        call = uni(callv) + 1u;
        const char* ib = myws + (call % NBUF) * PAR + W_OFF + cidx * IMG_B + wave * 8192;
        __amdgpu_buffer_rsrc_t ri = mkrs(ib, 65536 + 8192);
        const char* pp = ib + (lane & 2) * 32768 + (lane & 1) * 4096 + 4092;
        unsigned tries = 0;
        bool have;
        do {
            while (!all_set(ld_sys(pp) != 0u)) {
                if (expired(myws, call, t0, tries)) { bad = true; break; }
                __builtin_amdgcn_s_sleep(4);
            }
            unsigned m = 0xffffffffu;
            #pragma unroll
            for (int s = 0; s < 8; ++s) {
                const u4 a = cld(ri, (unsigned)(lane * 16 + (s & 1) * 2048), (s >> 2) * 65536 + ((s >> 1) & 1) * 4096);
                const u4 b = cld(ri, (unsigned)(lane * 16 + (s & 1) * 2048 + 1024), (s >> 2) * 65536 + ((s >> 1) & 1) * 4096);
                w0[s] = __builtin_bit_cast(h8, a); w1[s] = __builtin_bit_cast(h8, b);
                m = min(m, min(min4(a), min4(b)));
            }
            have = all_set(m != 0u);
            if (!have && expired(myws, call, t0, tries)) bad = true;
        } while (!have && !bad);
        if (bad) lds_st(abortp, 1u);
    }
    st_agent(ctrp, call);

    __amdgpu_buffer_rsrc_t rx = mkrs(myws + (call % NBUF) * PAR + (l * 16 + tp) * TILE_B + wave * 4096, 15 * TILE_B);

    auto process = [&](auto nt_tag, int ma, int mb) -> unsigned {
        constexpr int NT = decltype(nt_tag)::value;
        u4 a[NT * 8];
        int so[2] = {(int)uni(ma * (2 * TILE_B)), (int)uni(mb * (2 * TILE_B))};
        asm volatile("" : "+s"(so[0]), "+s"(so[1]));
        #pragma unroll
        for (int t = 0; t < NT; ++t) {
            #pragma unroll
            for (int s = 0; s < 8; ++s) {
                a[t * 8 + s] = cld(rx, (unsigned)(lane * 16 + (s & 3) * 1024), so[t] + (s >> 2) * 32768);
                if ((s & 3) == 3) __builtin_amdgcn_sched_barrier(0);
            }
        }
        unsigned okm = 0;
        #pragma unroll
        for (int t = 0; t < NT; ++t) {
            f4 n0 = {0.f, 0.f, 0.f, 0.f}, n1 = {0.f, 0.f, 0.f, 0.f};
            unsigned mnA = 0xffffffffu, mnB = 0xffffffffu;
            #pragma unroll
            for (int s = 0; s < 4; ++s) {
                const h8 av = __builtin_bit_cast(h8, a[t * 8 + s]);
                n0 = __builtin_amdgcn_mfma_f32_16x16x32_f16(w0[s], av, n0, 0, 0, 0);
                n1 = __builtin_amdgcn_mfma_f32_16x16x32_f16(w1[s], av, n1, 0, 0, 0);
                mnA = min(mnA, min4(a[t * 8 + s]));
            }
            #pragma unroll
            for (int s = 4; s < 8; ++s) mnB = min(mnB, min4(a[t * 8 + s]));
            const bool okB = all_set(mnB != 0u);
            if (okB) {
                #pragma unroll
                for (int s = 4; s < 8; ++s) {
                    const h8 av = __builtin_bit_cast(h8, a[t * 8 + s]);
                    n0 = __builtin_amdgcn_mfma_f32_16x16x32_f16(w0[s], av, n0, 0, 0, 0);
                    n1 = __builtin_amdgcn_mfma_f32_16x16x32_f16(w1[s], av, n1, 0, 0, 0);
                }
            }
            const int m = t ? mb : ma;
            *(f4*)(myreg + m * 2048 + lane * 16) = n0;
            *(f4*)(myreg + m * 2048 + 1024 + lane * 16) = n1;
            if (all_set(mnA != 0u)) okm |= (okB ? 1u : 256u) << m;
            __builtin_amdgcn_sched_barrier(0);
        }
        return okm;
    };
    auto processB = [&](int m) -> unsigned {
        u4 a[4];
        int so = (int)uni(m * (2 * TILE_B) + 32768);
        asm volatile("" : "+s"(so));
        #pragma unroll
        for (int s = 0; s < 4; ++s) a[s] = cld(rx, (unsigned)(lane * 16 + s * 1024), so);
        f4 n0 = *(const f4*)(myreg + m * 2048 + lane * 16), n1 = *(const f4*)(myreg + m * 2048 + 1024 + lane * 16);
        __builtin_amdgcn_sched_barrier(0);
        unsigned mn = 0xffffffffu;
        #pragma unroll
        for (int s = 0; s < 4; ++s) {
            const h8 av = __builtin_bit_cast(h8, a[s]);
            n0 = __builtin_amdgcn_mfma_f32_16x16x32_f16(w0[4 + s], av, n0, 0, 0, 0);
            n1 = __builtin_amdgcn_mfma_f32_16x16x32_f16(w1[4 + s], av, n1, 0, 0, 0);
            mn = min(mn, min4(a[s]));
        }
        if (!all_set(mn != 0u)) return 0u;
        *(f4*)(myreg + m * 2048 + lane * 16) = n0;
        *(f4*)(myreg + m * 2048 + 1024 + lane * 16) = n1;
        return 1u << m;
    };

    {
        unsigned pend = bad ? 0u : tmask, pendB = 0u, tries = 0;
        bool lastB = false;
        while (pend | pendB) {
            unsigned okm = 0;
            if (late(tries)) { bad = true; break; }
            unsigned mk = (unsigned)__ballot((lds_ld(rdy + (lane & 7)) >> wave) & 1u) & pend;
            if (pendB && !lastB) mk = 0;
            lastB = mk == 0u;
            if (mk) {
                const int ma = __builtin_ctz(mk); mk &= mk - 1;
                __builtin_amdgcn_s_setprio(2);
                if (mk) okm = process(std::integral_constant<int, 2>{}, ma, __builtin_ctz(mk));
                else okm = process(std::integral_constant<int, 1>{}, ma, ma);
                pend &= ~(okm | (okm >> 8));
                pendB |= okm >> 8;
                okm &= 0xFFu;
            } else if (pendB) {
                __builtin_amdgcn_s_setprio(2);
                unsigned t = pendB;
                do { const int m = __builtin_ctz(t); t &= t - 1; okm |= processB(m); } while (t);
                pendB &= ~okm;
            } else { __builtin_amdgcn_s_sleep(1); continue; }
            asm volatile("s_waitcnt lgkmcnt(0)" ::: "memory");
            if (lane < 8 && ((okm >> lane) & 1u)) __hip_atomic_fetch_add(cnt + lane, 1u, __ATOMIC_RELAXED, __HIP_MEMORY_SCOPE_WORKGROUP);
            __builtin_amdgcn_s_setprio(0);
        }
    }
    if (!bad && !foreign) {
        const int ti = 2 * wave + tp;
        const int srcr = ((ti < N1) ? gr ^ 1 : gr) * 4 + l;
        __amdgpu_buffer_rsrc_t ro = mkrs(p.out, OUT_BYTES);
        const unsigned om = (unsigned)(((lane >> 3) * NL) * 4 + (lane & 7) * 16);
        const unsigned so = (unsigned)(((srcr * MS + ti * 16) * NL + ncol) * 4);
        unsigned tries = 0;
        u4 a0 = cld(ro, om, so), b0 = cld(ro, om, so + 8 * NL * 4);
        __builtin_amdgcn_s_sleep(5);
        u4 a1 = cld(ro, om, so), b1 = cld(ro, om, so + 8 * NL * 4);
        __builtin_amdgcn_s_sleep(5);
        for (;;) {
            u4 a2 = cld(ro, om, so), b2 = cld(ro, om, so + 8 * NL * 4);
            __builtin_amdgcn_sched_barrier(0);
            if (all_set(max(max4(a0), max4(b0)) != SENT)) break;
            a0 = cld(ro, om, so); b0 = cld(ro, om, so + 8 * NL * 4);
            __builtin_amdgcn_sched_barrier(0);
            if (all_set(max(max4(a1), max4(b1)) != SENT)) break;
            a1 = cld(ro, om, so); b1 = cld(ro, om, so + 8 * NL * 4);
            __builtin_amdgcn_sched_barrier(0);
            if (all_set(max(max4(a2), max4(b2)) != SENT)) break;
            if (expired(myws, call, t0, tries)) break;
        }
    }
}

extern "C" void kernel_launch(int rank, int nranks, ncclComm_t comm,
                              hipStream_t stream,
                              void* const* d_in, const int* in_sizes, int n_in,
                              void* d_out, int out_size,
                              const int* in_dtypes, int out_dtype,
                              void* const* d_out_all,
                              void* const* d_ws_all, size_t ws_bytes) {
    if (nranks != NR || ws_bytes < WS_NEED || n_in < 2 || rank < 0 || rank >= NR ||
        in_sizes[0] != MS * KD || in_sizes[1] != KD * NL || out_size != NR * MS * NL) {
        fprintf(stderr, "unexpected configuration\n");
        return;
    }
    static bool init = false;
    if (!init) {
        HIP_CHECK(hipFuncSetAttribute((const void*)ag_gemm_kernel, hipFuncAttributeMaxDynamicSharedMemorySize, LDS_BYTES));
        init = true;
    }
    Params p;
    const int gr = rank >> 2, li = rank & 3;
    p.x = (const float*)d_in[0];
    p.w = (const float*)d_in[1];
    p.myws = (char*)d_ws_all[rank];
    p.rank = rank;
    p.pad = 0;
    p.out = (float*)d_out;
    for (int j = 0; j < 6; ++j) p.mates[j] = (char*)d_ws_all[gr * 4 + ((li + 1 + j % 3) & 3)];
    for (int j = 0; j < 8; ++j) p.opps[j] = (char*)d_ws_all[(gr ^ 1) * 4 + (j & 3)];
    for (int j = 0; j < 4; ++j) p.oout[j] = (float*)d_out_all[(gr ^ 1) * 4 + j];
    hipLaunchKernelGGL(ag_gemm_kernel, dim3(GRID), dim3(BLOCK), LDS_BYTES, stream, p);
    HIP_CHECK(hipGetLastError());
}
```

```cpp
#include <hip/hip_runtime.h>
#include <rccl/rccl.h>
#include <cstdio>
#include <type_traits>

#define HIP_CHECK(expr) do { hipError_t _e = (expr); if (_e != hipSuccess) { \
    fprintf(stderr, "HIP error %s at %s:%d\n", hipGetErrorString(_e), __FILE__, __LINE__); \
    return; } } while (0)

typedef float f4 __attribute__((ext_vector_type(4)));
typedef unsigned u4 __attribute__((ext_vector_type(4)));
typedef _Float16 h8 __attribute__((ext_vector_type(8)));
typedef __attribute__((address_space(1))) unsigned gu32;
typedef __attribute__((address_space(3))) unsigned lu32;

constexpr int N1 = 12;
constexpr int N2 = 16 - N1;
constexpr int NR = 8;
constexpr int KD = 2048;
constexpr int MS = 256;
constexpr int NL = 256;
constexpr unsigned TILE_B = 16 * KD * 2;
constexpr unsigned X_BYTES = 64 * TILE_B;
constexpr unsigned IMG_B = KD * 32 * 2;
constexpr unsigned W_OFF = X_BYTES;
constexpr unsigned PAR = X_BYTES + 8 * IMG_B;
constexpr unsigned NBUF = 3;
constexpr unsigned CTR_OFF = NBUF * PAR;
constexpr unsigned PFLAG_OFF = CTR_OFF + 256 * 64;
constexpr unsigned RFLAG_OFF = PFLAG_OFF + 256;
constexpr unsigned TMO_OFF = RFLAG_OFF + 512;
constexpr unsigned WS_NEED = TMO_OFF + 256;
constexpr unsigned OUT_BYTES = 2048u * NL * 4;
constexpr unsigned SENT = 0xFFFFFFFFu;
constexpr unsigned QNAN = 0x7FC00000u;
constexpr int GRID = 256;
constexpr int BLOCK = 768;
constexpr unsigned PART_B = 8 * 16384;
constexpr int LDS_BYTES = PART_B + 128;
constexpr unsigned DEADLINE = 50000000u;
constexpr float WSCALE = 64.f, WINV = 1.f / 64.f;

constexpr unsigned IT0 = 45;
constexpr unsigned ITP = 58;
constexpr unsigned WTP = 56;
constexpr unsigned GT = 700;
constexpr unsigned FSP = 60;
constexpr unsigned HKT = 100;
constexpr unsigned ZT0 = 150, ZTP = 25;
constexpr int IWIN = 8, WWIN = 8, XWIN = 4;

struct Params {
    const float* x;
    const float* w;
    char* myws;
    int rank;
    int pad;
    float* out;
    char* mates[6];
    char* opps[8];
    float* oout[4];
};

__device__ __forceinline__ unsigned uni(unsigned v) { return __builtin_amdgcn_readfirstlane(v); }
__device__ __forceinline__ unsigned now32() { return (unsigned)__builtin_amdgcn_s_memrealtime(); }
__device__ __forceinline__ unsigned ld_agent(const char* p) {
    return __hip_atomic_load((gu32*)p, __ATOMIC_RELAXED, __HIP_MEMORY_SCOPE_AGENT);
}
__device__ __forceinline__ unsigned ld_sys(const char* p) {
    return __hip_atomic_load((gu32*)p, __ATOMIC_RELAXED, __HIP_MEMORY_SCOPE_SYSTEM);
}
__device__ __forceinline__ void st_agent(char* p, unsigned v) {
    __hip_atomic_store((gu32*)p, v, __ATOMIC_RELAXED, __HIP_MEMORY_SCOPE_AGENT);
}
__device__ __forceinline__ unsigned lds_ld(lu32* a) { return __hip_atomic_load(a, __ATOMIC_RELAXED, __HIP_MEMORY_SCOPE_WORKGROUP); }
__device__ __forceinline__ void lds_st(lu32* a, unsigned v) { __hip_atomic_store(a, v, __ATOMIC_RELAXED, __HIP_MEMORY_SCOPE_WORKGROUP); }
__device__ __forceinline__ __amdgpu_buffer_rsrc_t mkrs(const void* p, unsigned bytes) {
    const unsigned long long a = (unsigned long long)p;
    const unsigned lo = uni((unsigned)a), hi = uni((unsigned)(a >> 32));
    return __builtin_amdgcn_make_buffer_rsrc((void*)(((unsigned long long)hi << 32) | lo), 0, bytes, 0x00020000);
}
__device__ __forceinline__ f4 bld(__amdgpu_buffer_rsrc_t r, unsigned vo, unsigned so) {
    return __builtin_bit_cast(f4, __builtin_amdgcn_raw_buffer_load_b128(r, vo, so, 0));
}
__device__ __forceinline__ u4 cld(__amdgpu_buffer_rsrc_t r, unsigned vo, unsigned so) {
    return __builtin_amdgcn_raw_buffer_load_b128(r, vo, so, 17);
}
__device__ __forceinline__ void cst(u4 v, __amdgpu_buffer_rsrc_t r, unsigned vo, unsigned so) {
    __builtin_amdgcn_raw_buffer_store_b128(v, r, vo, so, 17);
}
__device__ __forceinline__ u4 nz(u4 v) {
    #pragma unroll
    for (int e = 0; e < 4; ++e) v[e] = v[e] == 0u ? 0x80000000u : v[e];
    return v;
}
__device__ __forceinline__ u4 cvt8(f4 a, f4 b) {
    h8 h;
    h[0] = (_Float16)a[0]; h[1] = (_Float16)a[1]; h[2] = (_Float16)a[2]; h[3] = (_Float16)a[3];
    h[4] = (_Float16)b[0]; h[5] = (_Float16)b[1]; h[6] = (_Float16)b[2]; h[7] = (_Float16)b[3];
    return nz(__builtin_bit_cast(u4, h));
}
__device__ __forceinline__ unsigned min4(u4 v) { return min(min(v[0], v[1]), min(v[2], v[3])); }
__device__ __forceinline__ unsigned max4(u4 v) { return max(max(v[0], v[1]), max(v[2], v[3])); }
__device__ __forceinline__ bool all_set(bool b) { return __ballot(b) == ~0ull; }
__device__ __forceinline__ void until(unsigned t0, unsigned ticks) {
    while (now32() - t0 < ticks) __builtin_amdgcn_s_sleep(1);
}
template <int N> __device__ __forceinline__ void vm_wait() { asm volatile("s_waitcnt vmcnt(%0)" :: "i"(N) : "memory"); }

__device__ __forceinline__ float silu(float y) {
    const float c = fminf(fmaxf(y, -60.f), 60.f);
    return y * __builtin_amdgcn_rcpf(1.f + __expf(-c));
}

__device__ __forceinline__ bool expired(char* myws, unsigned call, unsigned t0, unsigned& tries) {
    tries = uni(tries + 1u);
    if ((tries & 31u) != 0u) return false;
    if (uni(ld_agent(myws + TMO_OFF)) == call || now32() - t0 > DEADLINE) {
        st_agent(myws + TMO_OFF, call);
        return true;
    }
    return false;
}

__device__ __forceinline__ void w_to_lds(const f4 (&r)[8], char* lw, int sw) {
    #pragma unroll
    for (int i = 0; i < 4; ++i) {
        h8 h;
        #pragma unroll
        for (int e = 0; e < 8; ++e) h[e] = (_Float16)(r[e][i] * WSCALE);
        *(h8*)(lw + ((i ^ sw) * 16)) = h;
    }
}

__global__ void __launch_bounds__(BLOCK, 3) ag_gemm_kernel(Params p) {
    extern __shared__ __attribute__((aligned(16))) char smem[];
    const int lane = threadIdx.x & 63;
    const int wave = uni(threadIdx.x >> 6);
    const int bid = blockIdx.x;
    const float* px = p.x;
    const float* pw = p.w;
    char* myws = p.myws;
    const int rank = p.rank;
    asm volatile("" :: "s"(px), "s"(pw), "s"(myws), "s"(rank));
    const int gr = rank >> 2, li = rank & 3;
    const unsigned t0 = now32();

    char* ctrp = myws + CTR_OFF + bid * 64 + wave * 4;
    const int q = lane & 7, kgl = lane >> 3;
    const int lwo = (((kgl >> 2) * 2 + (q >> 2)) * 64 + (kgl & 3) * 16 + 4 * (q & 3)) * 16;
    const int lr0 = ((lane >> 4) * 16 + ((lane & 15) ^ ((lane >> 3) & 1))) * 16;
    const int lr1 = ((lane >> 4) * 16 + ((lane & 15) ^ (((lane >> 3) & 1) | 2))) * 16 + 1024;

    if (__builtin_expect(bid < 32 && wave == 0, 1)) {
        const int j = bid;
        __amdgpu_buffer_rsrc_t rxin = mkrs(px, MS * KD * 4);
        const unsigned xv = (unsigned)(((lane & 15) * KD + (lane >> 4) * 8 + j * 32) * 4);
        constexpr int NU = 2 * N1, PB = 12;
        f4 xa[PB], xb[PB];
        unsigned callv = 0;
        #pragma unroll
        for (int u = 0; u < PB; ++u) {
            xa[u] = bld(rxin, xv, (u >> 1) * 16 * KD * 4 + (u & 1) * 4096); xb[u] = bld(rxin, xv + 16, (u >> 1) * 16 * KD * 4 + (u & 1) * 4096);
            if (u == 0) { __builtin_amdgcn_sched_barrier(0); callv = ld_agent(ctrp); __builtin_amdgcn_sched_barrier(0); }
        }
        __amdgpu_buffer_rsrc_t rs[4];
        const int rot = j % 3;
        #pragma unroll
        for (int d = 0; d < 3; ++d) rs[d] = mkrs(p.mates[rot + d], NBUF * PAR);
        rs[3] = mkrs(myws, NBUF * PAR);
        __builtin_amdgcn_sched_barrier(0);
        const unsigned call = uni(callv) + 1u;
        const unsigned boff = (call % NBUF) * PAR + li * 16 * TILE_B + j * 1024;
        const unsigned voff = lane * 16;
        #pragma unroll
        for (int u = 0; u < NU; ++u) {
            const u4 v = cvt8(xa[u % PB], xb[u % PB]);
            if (u >= 2) until(t0, IT0 + (u - 1) * ITP);
            #pragma unroll
            for (int d = 0; d < 4; ++d) cst(v, rs[d], voff, boff + (u >> 1) * TILE_B + (u & 1) * 32768);
            if (u + PB < NU) {
                const unsigned so = ((u + PB) >> 1) * 16 * KD * 4 + ((u + PB) & 1) * 4096;
                xa[u % PB] = bld(rxin, xv, so); xb[u % PB] = bld(rxin, xv + 16, so);
            }
            if (u >= 1) asm volatile("s_waitcnt vmcnt(%0)" :: "i"(IWIN + (u + PB - 1 < NU ? 2 : 0) + (u + PB < NU ? 2 : 0)) : "memory");
            __builtin_amdgcn_sched_barrier(0);
        }
        st_agent(ctrp, call);
        return;
    }
    if (__builtin_expect(bid < 64 && wave == 0, 1)) {
        const int jj = bid - 32;
        const int dq = jj >> 3, kq = jj & 7;
        __amdgpu_buffer_rsrc_t rwin = mkrs(pw, KD * NL * 4);
        const unsigned wv = (unsigned)(((kq * 64 + kgl * 8) * NL + dq * 64 + q * 4) * 4);
        f4 r[4][8];
        auto wload = [&](int m) {
            #pragma unroll
            for (int e = 0; e < 8; ++e) r[m & 3][e] = bld(rwin, wv + (e & 3) * 1024, (m & 3) * 524288 + (m >> 2) * 128 + (e >> 2) * 4096);
        };
        wload(0);
        __builtin_amdgcn_sched_barrier(0);
        const unsigned callv = ld_agent(ctrp);
        __builtin_amdgcn_sched_barrier(0);
        #pragma unroll
        for (int m = 1; m < 4; ++m) wload(m);
        __amdgpu_buffer_rsrc_t rs[4];
        #pragma unroll
        for (int d = 0; d < 4; ++d) rs[d] = mkrs(p.opps[dq + d], NBUF * PAR);
        __builtin_amdgcn_sched_barrier(0);
        const unsigned call = uni(callv) + 1u;
        const unsigned par = (call % NBUF) * PAR;
        const unsigned woff = par + W_OFF + li * 2 * IMG_B + kq * 4096;
        const unsigned voff = lane * 16;
        auto wpark = [&](int m) { w_to_lds(r[m & 3], smem + m * 4096 + lwo, q >> 1); };
        auto wsend = [&](int m) {
            u4 v[4];
            #pragma unroll
            for (int u = 0; u < 4; ++u) v[u] = nz(*(const u4*)(smem + m * 4096 + (u >> 1) * 2048 + ((u & 1) ? lr1 : lr0)));
            if (m >= 2) until(t0, IT0 + 20 + (m - 1) * WTP);
            #pragma unroll
            for (int u = 0; u < 4; ++u) cst(v[u], rs[0], voff, woff + (m >> 2) * IMG_B + (m & 3) * 32768 + u * 1024);
            __builtin_amdgcn_sched_barrier(0);
        };
        wpark(0); wsend(0);
        wpark(1); wsend(1);
        wpark(2); wpark(3);
        __builtin_amdgcn_sched_barrier(0);
        #pragma unroll
        for (int m = 4; m < 8; ++m) wload(m);
        __builtin_amdgcn_sched_barrier(0);
        wsend(2);
        #pragma unroll
        for (int m = 4; m < 8; ++m) wpark(m);
        __builtin_amdgcn_sched_barrier(0);
        __amdgpu_buffer_rsrc_t rxin = mkrs(px, MS * KD * 4);
        const unsigned xv = (unsigned)(((N1 * 16 + (lane & 15)) * KD + (lane >> 4) * 8 + jj * 32) * 4);
        f4 xa[2 * N2], xb[2 * N2];
        #pragma unroll
        for (int u = 0; u < 2 * N2; ++u) {
            xa[u] = bld(rxin, xv, (u >> 1) * 16 * KD * 4 + (u & 1) * 4096); xb[u] = bld(rxin, xv + 16, (u >> 1) * 16 * KD * 4 + (u & 1) * 4096);
        }
        __builtin_amdgcn_sched_barrier(0);
        wsend(3);
        wsend(4);
        #pragma unroll
        for (int m = 5; m < 8; ++m) { vm_wait<WWIN>(); wsend(m); }
        const unsigned boff = par + (li * 16 + N1) * TILE_B + jj * 1024;
        #pragma unroll
        for (int u = 0; u < 2 * N2; ++u) {
            const u4 v = cvt8(xa[u], xb[u]);
            #pragma unroll
            for (int d = 0; d < 4; ++d) cst(v, rs[d], voff, boff + (u >> 1) * TILE_B + (u & 1) * 32768);
            vm_wait<XWIN>();
            __builtin_amdgcn_sched_barrier(0);
        }
        st_agent(ctrp, call);
        return;
    }
    if (bid < 64) {
        if (wave > 2 || (wave == 2 && bid != 31)) return;
        const unsigned call = uni(ld_agent(ctrp)) + 1u;
        st_agent(ctrp, call);
        if (wave == 2) {
            unsigned tries = 0;
            while (!all_set(ld_agent(myws + PFLAG_OFF + lane * 4) == call)) {
                if (expired(myws, call, t0, tries)) return;
                __builtin_amdgcn_s_sleep(2);
            }
            if (lane < 4)
                __hip_atomic_store((gu32*)(p.opps[lane] + RFLAG_OFF + li * 128), call, __ATOMIC_RELAXED, __HIP_MEMORY_SCOPE_SYSTEM);
            return;
        }
        if (wave == 1) {
            const int ti = bid & 15;
            __amdgpu_buffer_rsrc_t ro = mkrs(p.out, OUT_BYTES);
            const unsigned so = (unsigned)((((ti < N1 ? gr ^ 1 : gr) * 4 + (bid >> 4)) * MS + ti * 16) * NL * 4);
            const u4 sv = {SENT, SENT, SENT, SENT};
            until(t0, HKT);
            #pragma unroll
            for (int e = 0; e < 16; ++e) cst(sv, ro, (unsigned)(lane * 16), so + e * 1024);
            vm_wait<0>();
            st_agent(myws + PFLAG_OFF + bid * 4, call);
        }
        return;
    }

    const int xg = bid & 7;
    const bool foreign = bid < 192;
    const int cidx = foreign ? (bid - 64) >> 4 : 8 + ((bid - 192) >> 3);
    const int fpar = (bid >> 3) & 1;
    const unsigned tmask = foreign ? (0x55u << fpar) : 0xFFu;
    const int mstep = foreign ? 2 : 1;
    const unsigned callv = ld_agent(ctrp);
    lu32* cnt = (lu32*)(smem + PART_B);
    lu32* rdy = cnt + 8;
    lu32* abortp = cnt + 16;
    const int l = xg >> 1, tp = xg & 1;
    const int ncol = foreign ? li * 64 + (cidx & 1) * 32 : (cidx - 8) * 32;
    constexpr int m2 = N1 / 2;
    static_assert(N1 % 2 == 0 && m2 % 2 == 0, "tile streams");
    auto late = [&](unsigned& tries) -> bool {
        tries = uni(tries + 1u);
        if ((tries & 127u) != 0u) return false;
        return uni(lds_ld(abortp)) != 0u || now32() - t0 > DEADLINE + 1000000u;
    };

    if (wave == 11) {
        __builtin_amdgcn_s_barrier();
        if (!foreign) return;
        const unsigned call = uni(callv) + 1u;
        st_agent(ctrp, call);
        constexpr unsigned ZB = PAR / 128;
        __amdgpu_buffer_rsrc_t rz = mkrs(myws + ((call + 2u) % NBUF) * PAR + (bid - 64) * ZB, ZB);
        const u4 zero = {0u, 0u, 0u, 0u};
        const unsigned ph = (unsigned)((bid * 29) % 32) * ZTP / 32;
        #pragma unroll 1
        for (unsigned u = 0; u < ZB / 1024; ++u) {
            until(t0, ZT0 + ph + u * ZTP);
            cst(zero, rz, (unsigned)(lane * 16), u * 1024);
        }
        return;
    }
    if (wave >= 8) {
        if (wave == 8) lds_st(cnt + (lane & 31), 0u);
        asm volatile("s_waitcnt lgkmcnt(0)" ::: "memory");
        __builtin_amdgcn_s_barrier();
        const unsigned call = uni(callv) + 1u;
        st_agent(ctrp, call);
        __builtin_amdgcn_s_setprio(3);
        if (wave == 8) {
            const char* xb = myws + (call % NBUF) * PAR + (l * 16 + tp) * TILE_B + 1020 + (lane & 31) * 1024;
            const bool hi = lane >= 32;
            int ma = foreign ? fpar : 0, mb = foreign ? m2 + fpar : m2;
            unsigned ka = 0, kb = 0, tries = 0;
            while (ma < m2 || mb < 8) {
                const unsigned oa = (unsigned)min(ma, m2 - 1) * (2 * TILE_B), ob = (unsigned)min(mb, 7) * (2 * TILE_B);
                const unsigned long long bal = __ballot(ld_sys(xb + (hi ? ob : oa)) != 0u);
                #pragma unroll
                for (int h = 0; h < 2; ++h) {
                    unsigned b = (unsigned)(bal >> (32 * h));
                    int& mm = h ? mb : ma; unsigned& kk = h ? kb : ka;
                    if (mm < (h ? 8 : m2)) {
                        b &= b >> 1; b &= b >> 2; b &= 0x11111111u;
                        b = (b | (b >> 3)) & 0x03030303u; b = (b | (b >> 6)) & 0x000F000Fu; b = (b | (b >> 12)) & 0xFFu;
                        if (b != kk) { kk = b; lds_st(rdy + mm, b); }
                        if (b == 0xFFu) { mm += mstep; kk = 0; }
                    }
                }
                if (expired(myws, call, t0, tries)) { lds_st(abortp, 1u); break; }
                if (bal == 0ull) __builtin_amdgcn_s_sleep(2);
            }
        } else {
            __amdgpu_buffer_rsrc_t ro = mkrs(foreign ? (const void*)p.oout[cidx >> 1] : (const void*)p.out, OUT_BYTES);
            const int fh = wave - 9;
            const int fro = ((lane >> 2) & 1) * 1024 + ((lane & 3) * 16 + fh * 8 + (lane >> 3)) * 16;
            const unsigned fso = (unsigned)((fh * 8 + (lane >> 3)) * NL * 4 + (lane & 7) * 16);
            unsigned pend = tmask, tries = 0, bad = 0;
            unsigned nxt = t0 + GT + (unsigned)(((bid * 2 + fh) * 37) % 64) * FSP / 64;
            if (foreign) {
                while (uni(ld_sys(myws + RFLAG_OFF + (cidx >> 1) * 128)) != call) {
                    if (expired(myws, call, t0, tries)) { lds_st(abortp, 1u); pend = 0u; break; }
                    __builtin_amdgcn_s_sleep(8);
                }
                tries = 0;
            }
            while (pend) {
                unsigned mk = (unsigned)__ballot(lds_ld(cnt + (lane & 7)) == 8u) & pend;
                if (!mk) {
                    if (late(tries)) { bad = 1; mk = pend; }
                    else { __builtin_amdgcn_s_sleep(1); continue; }
                }
                asm volatile("" ::: "memory");
                const int m = __builtin_ctz(mk);
                pend &= ~(1u << m);
                f4 pa[8];
                #pragma unroll
                for (int v = 0; v < 8; ++v) pa[v] = *(const f4*)(smem + v * 16384 + m * 2048 + fro);
                const f4 a0 = ((pa[0] + pa[1]) + (pa[2] + pa[3])) + ((pa[4] + pa[5]) + (pa[6] + pa[7]));
                u4 o0;
                #pragma unroll
                for (int r = 0; r < 4; ++r) {
                    o0[r] = __builtin_bit_cast(unsigned, silu(a0[r] * WINV));
                    o0[r] = (bad || o0[r] == SENT) ? QNAN : o0[r];
                }
                const int ti = 2 * m + tp;
                const int srcr = ((ti < N1) ? gr : (gr ^ 1)) * 4 + l;
                if (foreign) {
                    if (!pend) nxt = t0 + GT;
                    while ((int)(now32() - nxt) < 0) __builtin_amdgcn_s_sleep(1);
                    nxt = now32() + FSP;
                }
                cst(o0, ro, fso, (unsigned)(((srcr * MS + ti * 16) * NL + ncol) * 4));
                if (bad) tries = 127u;
            }
        }
        return;
    }

    char* myreg = smem + wave * 16384;
    h8 w0[8], w1[8];
    unsigned call;
    bool bad = false;
    if (!foreign) {
        __amdgpu_buffer_rsrc_t rwin = mkrs(pw, KD * NL * 4);
        const unsigned wv = (unsigned)(((wave * 128 + kgl * 8) * NL + ncol + q * 4) * 4);
        f4 r[4][8];
        #pragma unroll
        for (int b = 0; b < 4; ++b) {
            #pragma unroll
            for (int j = 0; j < 8; ++j) r[b][j] = bld(rwin, wv + (j & 3) * 1024, ((b >> 1) * 1024 + (b & 1) * 64) * NL * 4 + (j >> 2) * 4096);
        }
        __builtin_amdgcn_s_barrier();
        #pragma unroll
        for (int b = 0; b < 4; ++b) w_to_lds(r[b], myreg + b * 4096 + lwo, q >> 1);
        #pragma unroll
        for (int s = 0; s < 8; ++s) { w0[s] = *(const h8*)(myreg + s * 2048 + lr0); w1[s] = *(const h8*)(myreg + s * 2048 + lr1); }
        call = uni(callv) + 1u;
    } else {
        __builtin_amdgcn_s_barrier();
        call = uni(callv) + 1u;
        const char* ib = myws + (call % NBUF) * PAR + W_OFF + cidx * IMG_B + wave * 8192;
        __amdgpu_buffer_rsrc_t ri = mkrs(ib, 65536 + 8192);
        const char* pp = ib + (lane & 2) * 32768 + (lane & 1) * 4096 + 4092;
        unsigned tries = 0;
        bool have;
        do {
            while (!all_set(ld_sys(pp) != 0u)) {
                if (expired(myws, call, t0, tries)) { bad = true; break; }
                __builtin_amdgcn_s_sleep(4);
            }
            unsigned m = 0xffffffffu;
            #pragma unroll
            for (int s = 0; s < 8; ++s) {
                const u4 a = cld(ri, (unsigned)(lane * 16 + (s & 1) * 2048), (s >> 2) * 65536 + ((s >> 1) & 1) * 4096);
                const u4 b = cld(ri, (unsigned)(lane * 16 + (s & 1) * 2048 + 1024), (s >> 2) * 65536 + ((s >> 1) & 1) * 4096);
                w0[s] = __builtin_bit_cast(h8, a); w1[s] = __builtin_bit_cast(h8, b);
                m = min(m, min(min4(a), min4(b)));
            }
            have = all_set(m != 0u);
            if (!have && expired(myws, call, t0, tries)) bad = true;
        } while (!have && !bad);
        if (bad) lds_st(abortp, 1u);
    }
    st_agent(ctrp, call);
    const char* pfp = myws + PFLAG_OFF + (l * 16 + 2 * wave + tp) * 4;
    unsigned pfv = ld_agent(pfp);

    __amdgpu_buffer_rsrc_t rx = mkrs(myws + (call % NBUF) * PAR + (l * 16 + tp) * TILE_B + wave * 4096, 15 * TILE_B);

    auto process = [&](auto nt_tag, int ma, int mb) -> unsigned {
        constexpr int NT = decltype(nt_tag)::value;
        u4 a[NT * 8];
        int so[2] = {(int)uni(ma * (2 * TILE_B)), (int)uni(mb * (2 * TILE_B))};
        asm volatile("" : "+s"(so[0]), "+s"(so[1]));
        #pragma unroll
        for (int t = 0; t < NT; ++t) {
            #pragma unroll
            for (int s = 0; s < 8; ++s) {
                a[t * 8 + s] = cld(rx, (unsigned)(lane * 16 + (s & 3) * 1024), so[t] + (s >> 2) * 32768);
                if ((s & 3) == 3) __builtin_amdgcn_sched_barrier(0);
            }
        }
        unsigned okm = 0;
        #pragma unroll
        for (int t = 0; t < NT; ++t) {
            f4 n0 = {0.f, 0.f, 0.f, 0.f}, n1 = {0.f, 0.f, 0.f, 0.f};
            unsigned mnA = 0xffffffffu, mnB = 0xffffffffu;
            #pragma unroll
            for (int s = 0; s < 4; ++s) {
                const h8 av = __builtin_bit_cast(h8, a[t * 8 + s]);
                n0 = __builtin_amdgcn_mfma_f32_16x16x32_f16(w0[s], av, n0, 0, 0, 0);
                n1 = __builtin_amdgcn_mfma_f32_16x16x32_f16(w1[s], av, n1, 0, 0, 0);
                mnA = min(mnA, min4(a[t * 8 + s]));
            }
            #pragma unroll
            for (int s = 4; s < 8; ++s) mnB = min(mnB, min4(a[t * 8 + s]));
            const bool okB = all_set(mnB != 0u);
            if (okB) {
                #pragma unroll
                for (int s = 4; s < 8; ++s) {
                    const h8 av = __builtin_bit_cast(h8, a[t * 8 + s]);
                    n0 = __builtin_amdgcn_mfma_f32_16x16x32_f16(w0[s], av, n0, 0, 0, 0);
                    n1 = __builtin_amdgcn_mfma_f32_16x16x32_f16(w1[s], av, n1, 0, 0, 0);
                }
            }
            const int m = t ? mb : ma;
            *(f4*)(myreg + m * 2048 + lane * 16) = n0;
            *(f4*)(myreg + m * 2048 + 1024 + lane * 16) = n1;
            if (all_set(mnA != 0u)) okm |= (okB ? 1u : 256u) << m;
            __builtin_amdgcn_sched_barrier(0);
        }
        return okm;
    };
    auto processB = [&](int m) -> unsigned {
        u4 a[4];
        int so = (int)uni(m * (2 * TILE_B) + 32768);
        asm volatile("" : "+s"(so));
        #pragma unroll
        for (int s = 0; s < 4; ++s) a[s] = cld(rx, (unsigned)(lane * 16 + s * 1024), so);
        f4 n0 = *(const f4*)(myreg + m * 2048 + lane * 16), n1 = *(const f4*)(myreg + m * 2048 + 1024 + lane * 16);
        __builtin_amdgcn_sched_barrier(0);
        unsigned mn = 0xffffffffu;
        #pragma unroll
        for (int s = 0; s < 4; ++s) {
            const h8 av = __builtin_bit_cast(h8, a[s]);
            n0 = __builtin_amdgcn_mfma_f32_16x16x32_f16(w0[4 + s], av, n0, 0, 0, 0);
            n1 = __builtin_amdgcn_mfma_f32_16x16x32_f16(w1[4 + s], av, n1, 0, 0, 0);
            mn = min(mn, min4(a[s]));
        }
        if (!all_set(mn != 0u)) return 0u;
        *(f4*)(myreg + m * 2048 + lane * 16) = n0;
        *(f4*)(myreg + m * 2048 + 1024 + lane * 16) = n1;
        return 1u << m;
    };

    {
        unsigned pend = bad ? 0u : tmask, pendB = 0u, tries = 0;
        bool lastB = false;
        while (pend | pendB) {
            unsigned okm = 0;
            if (late(tries)) { bad = true; break; }
            unsigned mk = (unsigned)__ballot((lds_ld(rdy + (lane & 7)) >> wave) & 1u) & pend;
            if (pendB && !lastB) mk = 0;
            lastB = mk == 0u;
            if (mk) {
                const int ma = __builtin_ctz(mk); mk &= mk - 1;
                __builtin_amdgcn_s_setprio(2);
                if (mk) okm = process(std::integral_constant<int, 2>{}, ma, __builtin_ctz(mk));
                else okm = process(std::integral_constant<int, 1>{}, ma, ma);
                pend &= ~(okm | (okm >> 8));
                pendB |= okm >> 8;
                okm &= 0xFFu;
            } else if (pendB) {
                __builtin_amdgcn_s_setprio(2);
                unsigned t = pendB;
                do { const int m = __builtin_ctz(t); t &= t - 1; okm |= processB(m); } while (t);
                pendB &= ~okm;
            } else { __builtin_amdgcn_s_sleep(1); continue; }
            asm volatile("s_waitcnt lgkmcnt(0)" ::: "memory");
            if (lane < 8 && ((okm >> lane) & 1u)) __hip_atomic_fetch_add(cnt + lane, 1u, __ATOMIC_RELAXED, __HIP_MEMORY_SCOPE_WORKGROUP);
            __builtin_amdgcn_s_setprio(0);
        }
    }
    if (!bad && !foreign) {
        const int ti = 2 * wave + tp;
        const int srcr = ((ti < N1) ? gr ^ 1 : gr) * 4 + l;
        __amdgpu_buffer_rsrc_t ro = mkrs(p.out, OUT_BYTES);
        const unsigned om = (unsigned)(((lane >> 3) * NL) * 4 + (lane & 7) * 16);
        const unsigned so = (unsigned)(((srcr * MS + ti * 16) * NL + ncol) * 4);
        unsigned tries = 0;
        while (uni(pfv) != call) {
            if (expired(myws, call, t0, tries)) { cst(u4{QNAN, QNAN, QNAN, QNAN}, ro, om, so); return; }
            __builtin_amdgcn_s_sleep(2);
            pfv = ld_agent(pfp);
        }
        u4 a0 = cld(ro, om, so), b0 = cld(ro, om, so + 8 * NL * 4);
        __builtin_amdgcn_s_sleep(5);
        u4 a1 = cld(ro, om, so), b1 = cld(ro, om, so + 8 * NL * 4);
        __builtin_amdgcn_s_sleep(5);
        for (;;) {
            u4 a2 = cld(ro, om, so), b2 = cld(ro, om, so + 8 * NL * 4);
            __builtin_amdgcn_sched_barrier(0);
            if (all_set(max(max4(a0), max4(b0)) != SENT)) break;
            a0 = cld(ro, om, so); b0 = cld(ro, om, so + 8 * NL * 4);
            __builtin_amdgcn_sched_barrier(0);
            if (all_set(max(max4(a1), max4(b1)) != SENT)) break;
            a1 = cld(ro, om, so); b1 = cld(ro, om, so + 8 * NL * 4);
            __builtin_amdgcn_sched_barrier(0);
            if (all_set(max(max4(a2), max4(b2)) != SENT)) break;
            if (expired(myws, call, t0, tries)) break;
        }
    }
}

extern "C" void kernel_launch(int rank, int nranks, ncclComm_t comm,
                              hipStream_t stream,
                              void* const* d_in, const int* in_sizes, int n_in,
                              void* d_out, int out_size,
                              const int* in_dtypes, int out_dtype,
                              void* const* d_out_all,
                              void* const* d_ws_all, size_t ws_bytes) {
    if (nranks != NR || ws_bytes < WS_NEED || n_in < 2 || rank < 0 || rank >= NR ||
        in_sizes[0] != MS * KD || in_sizes[1] != KD * NL || out_size != NR * MS * NL) {
        fprintf(stderr, "unexpected configuration\n");
        return;
    }
    static bool init = false;
    if (!init) {
        HIP_CHECK(hipFuncSetAttribute((const void*)ag_gemm_kernel, hipFuncAttributeMaxDynamicSharedMemorySize, LDS_BYTES));
        init = true;
    }
    Params p;
    const int gr = rank >> 2, li = rank & 3;
    p.x = (const float*)d_in[0];
    p.w = (const float*)d_in[1];
    p.myws = (char*)d_ws_all[rank];
    p.rank = rank;
    p.pad = 0;
    p.out = (float*)d_out;
    for (int j = 0; j < 6; ++j) p.mates[j] = (char*)d_ws_all[gr * 4 + ((li + 1 + j % 3) & 3)];
    for (int j = 0; j < 8; ++j) p.opps[j] = (char*)d_ws_all[(gr ^ 1) * 4 + (j & 3)];
    for (int j = 0; j < 4; ++j) p.oout[j] = (float*)d_out_all[(gr ^ 1) * 4 + j];
    hipLaunchKernelGGL(ag_gemm_kernel, dim3(GRID), dim3(BLOCK), LDS_BYTES, stream, p);
    HIP_CHECK(hipGetLastError());
}
```
